```python
import numpy as np

kernel_name = "gpu_ag_gemm_m4096_k4096_n16384_w8a8_dqsilu"


ABSMAX_RTOL = 0.001

COMPUTE_DTYPE = "int8"
EPILOGUE = "dequant_silu"
IN_QUANT = "int8"
OUT_QUANT = None
IN_DTYPES = {"x": "int8", "w_mat": "int8",
             "scale_x": "float32", "scale_w": "float32"}
WORLD_SIZE = 8

def setup_inputs():

    x = np.random.randint(-127, 128, size=(4096, 4096)).astype(np.int8)
    w_mat = np.random.randint(-127, 128, size=(4096, 16384)).astype(np.int8)
    scale_x = (0.02 * (1.0 + np.random.rand(1))).astype(np.float32)
    scale_w = (0.02 * (1.0 + np.random.rand(1))).astype(np.float32)
    return {"x": x, "w_mat": w_mat, "scale_x": scale_x, "scale_w": scale_w}

def reference(x, w_mat, scale_x, scale_w):
    acc = x.astype(np.float64) @ w_mat.astype(np.float64)
    y = (acc.astype(np.float32) * (scale_x * scale_w)).astype(np.float32)
    return (y / (1.0 + np.exp(-np.clip(y, -60.0, 60.0)))).astype(np.float32)
```

```cpp
#include <hip/hip_runtime.h>
#include <rccl/rccl.h>
#include <cstdio>
#include <cstdint>
#include <type_traits>

#define HIP_CHECK(expr) do { hipError_t _e = (expr); if (_e != hipSuccess) { \
    fprintf(stderr, "HIP error %s at %s:%d\n", hipGetErrorString(_e), __FILE__, __LINE__); \
    return; } } while (0)

namespace {

typedef int v2i __attribute__((ext_vector_type(2)));
typedef int v4i __attribute__((ext_vector_type(4)));
typedef unsigned v4u __attribute__((ext_vector_type(4)));
typedef float v4f __attribute__((ext_vector_type(4)));
typedef unsigned long long u64;
#define LAS __attribute__((address_space(3)))
#define GAS __attribute__((address_space(1)))
#define CAS __attribute__((address_space(4)))

constexpr int NR = 8;
constexpr int KD = 4096;
constexpr int NL = 2048;
constexpr int MS = 512;
constexpr int NBANDS = 32;
constexpr unsigned BAND_BYTES = 16 * KD;
constexpr unsigned SHARD_BYTES = MS * KD;
constexpr size_t SLOT_BYTES = (size_t)NR * SHARD_BYTES;
constexpr size_t FLAG_OFF = 2 * SLOT_BYTES;
constexpr size_t CTR_OFF = FLAG_OFF + 16384;
constexpr size_t TMO_OFF = CTR_OFF + 1024 * 64;
constexpr size_t DBG_OFF = TMO_OFF + 4096;
constexpr size_t WS_NEED = DBG_OFF + (1 << 19);
constexpr u64 DEADLINE_TICKS = 40000000ull;

constexpr int NITEMS = 6 + NBANDS;
constexpr int PUSH_SLOTS = 4;
constexpr int GRID = 8 * (PUSH_SLOTS + 28);
#define PUSH_W 16
#define POLL_S 2
#define AUX_X 17
#define AUX_OUT 16

struct Params {
    const int8_t* x; const int8_t* w; const float* sx; const float* sw; float* out;
    char* myws;
    char* peer[NR];
    int rank;
};

template <int I, int N, class F> __device__ __forceinline__ void static_for(F&& f) {
    if constexpr (I < N) { f(std::integral_constant<int, I>{}); static_for<I + 1, N>(f); }
}
template <int N> __device__ __forceinline__ void wait_vm() { static_assert(N >= 0 && N < 64, "vmcnt"); asm volatile("s_waitcnt vmcnt(%0)" :: "n"(N) : "memory"); }
__device__ __forceinline__ void wait_lgkm0() { asm volatile("s_waitcnt lgkmcnt(0)" ::: "memory"); }
__device__ __forceinline__ void wg_barrier() { asm volatile("" ::: "memory"); __builtin_amdgcn_s_barrier(); asm volatile("" ::: "memory"); }
#define SCHED_FENCE() __builtin_amdgcn_sched_barrier(0)

__device__ __forceinline__ __amdgpu_buffer_rsrc_t mk_rsrc(const void* p, unsigned bytes) {
    return __builtin_amdgcn_make_buffer_rsrc((void*)p, (short)0, (int)bytes, 0x00020000);
}
__device__ __forceinline__ v4i mk_rsrc_raw(const void* p, unsigned bytes) {
    const u64 a = (u64)p;
    return (v4i){(int)(unsigned)a, (int)((unsigned)(a >> 32) & 0xffffu), (int)bytes, 0x00020000};
}
__device__ __forceinline__ void dma16(v4i rsrc, int voff, int soff, unsigned lds_dst) {
    unsigned keep;
    asm volatile("s_mov_b32 %0, m0\n\ts_mov_b32 m0, %4\n\ts_nop 0\n\tbuffer_load_dwordx4 %1, %2, %3 offen lds\n\ts_mov_b32 m0, %0"
                 : "=&s"(keep) : "v"(voff), "s"(rsrc), "s"(soff), "s"(lds_dst) : "memory");
}

#define DBG_GB(gb) (gb)
#define DBG_WAIT true
#define STAMP(idx) do { } while (0)
#define STAMPC(idx) do { } while (0)

__device__ __forceinline__ void pusher(const int8_t* px, char* myws, int rank, int pj, int q, GAS unsigned* ctr, unsigned call,
                                       LAS char* smem, int lane, int wid) {
    constexpr int W = PUSH_W;
    static_assert(W >= 1 && W <= 16, "window");
    LAS volatile unsigned* done = (LAS volatile unsigned*)(smem + 65536);
    const CAS Params* kp = (const CAS Params*)__builtin_amdgcn_kernarg_segment_ptr();
    char* pws = kp->peer[(unsigned)pj & (NR - 1)];
    if (wid == 0) {
        const unsigned l0 = (unsigned)(size_t)smem;
        const int ss = q * 1024;
        const v4i rs = mk_rsrc_raw(px, SHARD_BYTES);
#define PUSH_DMA(b, r) dma16(rs, (lane ^ (r)) << 4, ss + (b) * (int)BAND_BYTES + (r) * KD, l0 + ((b) & 3) * 16384 + (r) * 1024)
#pragma unroll
        for (int b = 0; b < 3; ++b)
#pragma unroll
            for (int r = 0; r < 16; ++r) PUSH_DMA(b, r);
        if (lane == 0) *done = 0u;
        wait_lgkm0(); wg_barrier();
        const __amdgpu_buffer_rsrc_t rdst = mk_rsrc(pws + (size_t)(call & 1) * SLOT_BYTES + (size_t)rank * SHARD_BYTES, SHARD_BYTES);
        int rd[4];
#pragma unroll
        for (int i = 0; i < 4; ++i) rd[i] = (lane & 15) * 1024 + (((4 * i + (lane >> 4)) ^ (lane & 15)) << 4);
        const int vd = lane * 16, sd = q * 16384;
        v4u ta[16], tb[16];
#define PUSH_READ(t, b) do { _Pragma("unroll") for (int i = 0; i < 16; ++i) t[i] = *(LAS v4u*)(smem + ((b) & 3) * 16384 + rd[i & 3] + (i >> 2) * 256); } while (0)
#define PUSH_BAND(t, tn, b) do { \
            _Pragma("unroll") for (int i = 0; i < 16; ++i) { \
                wait_vm<2 * (W - 1)>(); \
                if (i == W - 1) { if (lane == 0) *done = (unsigned)(b); }        \
                if (i == 8) { PUSH_READ(tn, (b) + 1); }                           \
                PUSH_DMA((b) + 3, i); \
                __builtin_amdgcn_raw_buffer_store_b128(t[i], rdst, vd + 1024 * (i & 3), sd + (b) * (int)BAND_BYTES + 4096 * (i >> 2), 17); \
                SCHED_FENCE(); \
            } } while (0)
        wait_vm<32>();
        PUSH_READ(ta, 0);
        STAMP(2);
#pragma unroll 1
        for (int b = 0; b < NBANDS; b += 2) {
            PUSH_BAND(ta, tb, b);
            PUSH_BAND(tb, ta, b + 1);
        }
        wait_vm<0>();
        if (lane == 0) *done = (unsigned)NBANDS;
#undef PUSH_BAND
#undef PUSH_READ
#undef PUSH_DMA
    } else if (wid == 1) {
        GAS unsigned* flag = (GAS unsigned*)(pws + FLAG_OFF + (size_t)(rank * 4 + q) * 128);
        const unsigned fbase = call * 64u;
        wg_barrier();
        unsigned sent = 0;
        while (sent < (unsigned)NBANDS) {
            const unsigned d = __builtin_amdgcn_readfirstlane(*done);
            if (d > sent) {
                if (lane == 0) __hip_atomic_store(flag, fbase + d, __ATOMIC_RELAXED, __HIP_MEMORY_SCOPE_SYSTEM);
                sent = d;
            } else __builtin_amdgcn_s_sleep(1);
        }
    }
    if (lane == 0) __hip_atomic_store(ctr, call, __ATOMIC_RELAXED, __HIP_MEMORY_SCOPE_AGENT);
    STAMP(1);
}

__device__ __forceinline__ int poll_flags(u64 flaga, unsigned fbase) {
    const unsigned v = __hip_atomic_load((GAS unsigned*)flaga, __ATOMIC_RELAXED, __HIP_MEMORY_SCOPE_SYSTEM);
    return (int)(__builtin_amdgcn_readfirstlane(v) - fbase);
}
__device__ __forceinline__ int wait_flag(u64 flaga, GAS unsigned* tmo, unsigned call, int need, bool& dead, int lane) {
    int known = need;
    if (!dead) {
        const u64 t0 = __builtin_amdgcn_s_memrealtime();
        for (;;) {
            known = poll_flags(flaga, call * 64u);
            if (known >= need) break;
            const u64 el = __builtin_amdgcn_s_memrealtime() - t0;
            if (el > 20000ull) {
                if (el > DEADLINE_TICKS ||
                    __builtin_amdgcn_readfirstlane(__hip_atomic_load(tmo, __ATOMIC_RELAXED, __HIP_MEMORY_SCOPE_AGENT)) == call) {
                    if (lane == 0) __hip_atomic_store(tmo, call, __ATOMIC_RELAXED, __HIP_MEMORY_SCOPE_AGENT);
                    dead = true; break;
                }
                __builtin_amdgcn_s_sleep(8);
            }
            __builtin_amdgcn_s_sleep(1);
        }
    }
    return known;
}

struct Epi {
    LAS char* ex;
    int wr[4];
    int rd;
    int vout;
    float scale;
};

__device__ __forceinline__ v4f silu4(v4i sum, float scale, bool dead) {
    v4f o;
#pragma unroll
    for (int j = 0; j < 4; ++j) {
        const float y = (float)sum[j] * scale;
        const float yc = __builtin_amdgcn_fmed3f(y, -60.0f, 60.0f);
        const float e = __builtin_amdgcn_exp2f(yc * -1.4426950408889634f);
        o[j] = y * __builtin_amdgcn_rcpf(1.0f + e);
    }
    if (dead) o = (v4f){__builtin_nanf(""), __builtin_nanf(""), __builtin_nanf(""), __builtin_nanf("")};
    return o;
}

#define MV(nv) do { __builtin_amdgcn_sched_group_barrier(0x8, 1, 0); __builtin_amdgcn_sched_group_barrier(0x2 | 0x400, nv, 0); } while (0)
__device__ __forceinline__ void band_step(const v4i (&wf)[16][4], v4i (&cur)[16], v4i (&acc)[4], const v4i (&accp)[4],
                                          __amdgpu_buffer_rsrc_t rx, int vrem, int sn,
                                          const Epi& e, int par, __amdgpu_buffer_rsrc_t ro, int sprev, bool dead,
                                          __amdgpu_buffer_rsrc_t rf, int vflag, unsigned& polled) {
    v4i p0, p1, p2, p3, sum;
    v4f y, t, o;
    LAS char* ex = e.ex + par * 4096;
    static_for<0, 16>([&](auto ss) { constexpr int s = ss;
        if constexpr (s == 3) {
            wait_lgkm0(); wg_barrier();
            p0 = *(LAS v4i*)(ex + e.rd); p1 = *(LAS v4i*)(ex + e.rd + 32768);
        }
        if constexpr (s == 4) { p2 = *(LAS v4i*)(ex + e.rd + 65536); p3 = *(LAS v4i*)(ex + e.rd + 98304); }
#pragma unroll
        for (int n = 0; n < 4; ++n) {
            acc[n] = __builtin_amdgcn_mfma_i32_16x16x64_i8(wf[s][n], cur[s], s == 0 ? (v4i){0, 0, 0, 0} : acc[n], 0, 0, 0);
            if constexpr (s < 2) { if (n & 1) { *(LAS v4i*)(ex + e.wr[2 * s + (n >> 1)]) = accp[2 * s + (n >> 1)]; SCHED_FENCE(); } }
        }
        cur[s] = (v4i)__builtin_amdgcn_raw_buffer_load_b128(rx, vrem + 1024 * (s & 3), sn + 4096 * (s >> 2), AUX_X);
        if constexpr (s == POLL_S) polled = __builtin_amdgcn_raw_buffer_load_b32(rf, vflag, 0, 17);
        if constexpr (s == 6) { sum = (p0 + p1) + (p2 + p3); MV(2); MV(2); MV(2); MV(2); }
        if constexpr (s == 7) { for (int j = 0; j < 4; ++j) { y[j] = (float)sum[j]; asm volatile("" : "+v"(y[j])); y[j] *= e.scale; } MV(2); MV(2); MV(2); MV(2); }
        if constexpr (s == 8) { for (int j = 0; j < 4; ++j) t[j] = __builtin_amdgcn_fmed3f(y[j], -60.0f, 60.0f) * -1.4426950408889634f; MV(2); MV(2); MV(2); MV(2); }
        if constexpr (s == 9) { for (int j = 0; j < 4; ++j) t[j] = __builtin_amdgcn_exp2f(t[j]); MV(1); MV(1); MV(1); MV(1); }
        if constexpr (s == 10) { for (int j = 0; j < 4; ++j) { t[j] = 1.0f + t[j]; asm volatile("" : "+v"(t[j])); } MV(1); MV(1); MV(1); MV(1); }
        if constexpr (s == 11) { for (int j = 0; j < 4; ++j) t[j] = __builtin_amdgcn_rcpf(t[j]); MV(1); MV(1); MV(1); MV(1); }
        if constexpr (s == 12) { for (int j = 0; j < 4; ++j) { o[j] = y[j] * t[j]; asm volatile("" : "+v"(o[j])); o[j] = dead ? __builtin_nanf("") : o[j]; } MV(2); MV(2); MV(2); MV(2); }
        if constexpr (s == 13) { __builtin_amdgcn_raw_buffer_store_b128((v4u)o, ro, e.vout, sprev, AUX_OUT); }
        SCHED_FENCE();
    });
}
#undef MV

__global__ void __launch_bounds__(256, 1) ag_gemm_kernel(Params P) {
    __shared__ __attribute__((aligned(16))) char smem[131072];
    const int lane = threadIdx.x & 63;
    const int wid = __builtin_amdgcn_readfirstlane(threadIdx.x >> 6);
    const int xcd = blockIdx.x & 7, slot = blockIdx.x >> 3;
    char* const myws = P.myws;
    const int8_t* const px = P.x;
    const int rank = P.rank;
    STAMP(0);

    GAS unsigned* ctr = (GAS unsigned*)(myws + CTR_OFF + (size_t)(blockIdx.x * 4 + wid) * 64);
    const unsigned call = *(const CAS unsigned*)(u64)ctr + 1u;

    if (slot < PUSH_SLOTS) {
#define PUSH_MAP 0
        pusher(px, myws, rank, (xcd + PUSH_MAP * slot) & 7, slot, ctr, call, (LAS char*)smem, lane, wid);
        return;
    }
    const float sxv = P.sx[0], swv = P.sw[0];

    const int cs = slot - PUSH_SLOTS;
    const int cg = xcd * 4 + (cs & 3);
    const int g = cs >> 2;
    const int src = (rank + 1 + g) & (NR - 1);
    const int n0 = cg * 64;

    v4i wf[16][4];
    v4i xa[16], xb[16];
    LAS char* myl = (LAS char*)smem + wid * 32768;
    {
        const unsigned l0 = (unsigned)(size_t)myl;
        const int row = lane >> 2;
        const v4i rw = mk_rsrc_raw(P.w, KD * NL);
        const int c0 = (lane & 3) ^ ((row >> 2) & 1);
        const int v0 = row * NL + c0 * 16, v1 = row * NL + (c0 ^ 2) * 16;
        const int sw0 = wid * 1024 * NL + n0;
#define DMA_CHUNK(ch, buf) do { _Pragma("unroll") for (int i = 0; i < 16; ++i) \
            dma16(rw, (i & 1) ? v1 : v0, sw0 + ((ch) * 256 + i * 16) * NL, l0 + (buf) * 16384 + i * 1024); } while (0)
        const int gq = lane >> 4, q = (lane & 15) >> 1, p = lane & 1;
        const int f = ((q >> 2) & 1) | ((gq & 1) << 1);
        const int rb = (16 * gq + q) * 64 + 8 * p;
#define READ_CHUNK(ch, buf) do { _Pragma("unroll") for (int s = 0; s < 4; ++s) _Pragma("unroll") for (int n = 0; n < 4; ++n) { \
            LAS char* a = myl + (buf) * 16384 + s * 4096 + rb + 16 * (n ^ f); \
            v2i lo = __builtin_amdgcn_ds_read_tr8_b64_v2i32((LAS v2i*)a); \
            v2i hi = __builtin_amdgcn_ds_read_tr8_b64_v2i32((LAS v2i*)(a + 512)); \
            wf[(ch) * 4 + s][n] = (v4i){lo.x, lo.y, hi.x, hi.y}; } \
            wait_lgkm0(); SCHED_FENCE(); } while (0)
        DMA_CHUNK(0, 0); DMA_CHUNK(1, 1);
        wait_vm<16>(); READ_CHUNK(0, 0);
        DMA_CHUNK(2, 0);
        wait_vm<16>(); READ_CHUNK(1, 1);
        DMA_CHUNK(3, 1);
        wait_vm<16>(); READ_CHUNK(2, 0);
        wait_vm<0>();  READ_CHUNK(3, 1);
#undef DMA_CHUNK
#undef READ_CHUNK
    }
    STAMP(1);

    if (lane == 0) __hip_atomic_store(ctr, call, __ATOMIC_RELAXED, __HIP_MEMORY_SCOPE_AGENT);
    const char* stg = myws + (size_t)(call & 1) * SLOT_BYTES;
    const u64 flagR = (u64)(myws + FLAG_OFF + (size_t)(src * 4 + wid) * 128);
    const u64 flagL = (u64)(myws + FLAG_OFF + (size_t)(rank * 4 + wid) * 128);
    GAS unsigned* tmo = (GAS unsigned*)(myws + TMO_OFF);
    int knownR = 0, knownL = 0;
    bool dead = false;
    constexpr unsigned OUT_BYTES = (unsigned)((size_t)NR * MS * NL * 4);
    const __amdgpu_buffer_rsrc_t ro = mk_rsrc(P.out, OUT_BYTES);
    const __amdgpu_buffer_rsrc_t rx = mk_rsrc(stg, (unsigned)SLOT_BYTES);
    Epi e;
    e.ex = (LAS char*)smem;
    {
        const int r = lane & 15, q = lane >> 4;
#pragma unroll
        for (int n = 0; n < 4; ++n) e.wr[n] = wid * 32768 + r * 256 + 16 * ((4 * n + q) ^ (r & 7));
        const int rrow = 4 * wid + q;
        e.rd = rrow * 256 + 16 * (r ^ (rrow & 7));
        e.vout = (rrow * NL + n0 + 4 * r) * 4;
        e.scale = sxv * swv;
    }
    const int vrem = lane * 16;
    const __amdgpu_buffer_rsrc_t rf = mk_rsrc(myws + FLAG_OFF, 16384);
    const int vflag = (((lane & 1) ? rank : src) * 4 + wid) * 128;
    const unsigned fbase = call * 64u;
    unsigned polled;
#define TAKE_POLL() do { \
        knownR = max(knownR, (int)(__builtin_amdgcn_readlane(polled, 0) - fbase)); \
        knownL = max(knownL, (int)(__builtin_amdgcn_readlane(polled, 1) - fbase)); } while (0)
    const int nit = g < 2 ? NBANDS + 6 : NBANDS + 4;
    int tab;
    {
        constexpr u64 lm4 = (1ull << 5) | (1ull << 11) | (1ull << 17) | (1ull << 22), lm6 = lm4 | (1ull << 25) | (1ull << 28);
        const u64 lmask = g < 2 ? lm6 : lm4;
        const int nl = __builtin_popcountll(lmask & ((1ull << lane) - 1ull));
        const int bl = nl < 4 ? g + 7 * nl : 24 + 2 * g + nl;
        tab = ((lmask >> lane) & 1ull) ? rank * NBANDS + bl : src * NBANDS + (lane - nl);
        tab = lane < nit ? tab : NR * NBANDS;
    }
#define ITEM_GB(i) __builtin_amdgcn_readlane(tab, (i))
#define ITEM_READY(i, sn) do { \
        const int gb_ = ITEM_GB(i); \
        const bool loc_ = (gb_ >> 5) == rank; \
        const int have_ = loc_ ? knownL : knownR; \
        if (DBG_WAIT && have_ <= (gb_ & 31) && (i) < nit) { \
            const int k_ = wait_flag(loc_ ? flagL : flagR, tmo, call, (gb_ & 31) + 1, dead, lane); \
            if (loc_) knownL = k_; else knownR = k_; \
        } \
        sn = DBG_GB(gb_) * (int)BAND_BYTES + wid * 16384; \
        asm volatile("" ::: "memory"); } while (0)

    {
        int sn;
        ITEM_READY(0, sn);
#pragma unroll
        for (int s = 0; s < 16; ++s) { xa[s] = (v4i)__builtin_amdgcn_raw_buffer_load_b128(rx, vrem + 1024 * (s & 3), sn + 4096 * (s >> 2), AUX_X); SCHED_FENCE(); }
        ITEM_READY(1, sn);
#pragma unroll
        for (int s = 0; s < 16; ++s) { xb[s] = (v4i)__builtin_amdgcn_raw_buffer_load_b128(rx, vrem + 1024 * (s & 3), sn + 4096 * (s >> 2), AUX_X); SCHED_FENCE(); }
    }
    STAMP(60); STAMPC(61);
    int sprev = (int)OUT_BYTES;
    v4i acca[4], accb[4];
#pragma unroll
    for (int n = 0; n < 4; ++n) accb[n] = (v4i){0, 0, 0, 0};
#pragma unroll 1
    for (int i = 0; i < nit; i += 2) {
        int sn;
        ITEM_READY(i + 2, sn);
        band_step(wf, xa, acca, accb, rx, vrem, sn, e, 1, ro, sprev, dead, rf, vflag, polled);
        TAKE_POLL();
        sprev = ITEM_GB(i) * (16 * NL * 4);
        STAMP(2 + i);
        ITEM_READY(i + 3, sn);
        band_step(wf, xb, accb, acca, rx, vrem, sn, e, 0, ro, sprev, dead, rf, vflag, polled);
        TAKE_POLL();
        sprev = ITEM_GB(i + 1) * (16 * NL * 4);
        STAMP(3 + i);
    }
    STAMP(62); STAMPC(63);
#undef ITEM_READY
#undef ITEM_GB
#undef TAKE_POLL
    {
        LAS char* ex = e.ex + 4096;
#pragma unroll
        for (int n = 0; n < 4; ++n) *(LAS v4i*)(ex + e.wr[n]) = accb[n];
        wait_lgkm0(); wg_barrier();
        LAS char* ep = ex + e.rd;
        const v4i sum = (*(LAS v4i*)(ep) + *(LAS v4i*)(ep + 32768)) + (*(LAS v4i*)(ep + 65536) + *(LAS v4i*)(ep + 98304));
        __builtin_amdgcn_raw_buffer_store_b128((v4u)silu4(sum, e.scale, dead), ro, e.vout, sprev, AUX_OUT);
        STAMP(2 + NITEMS);
    }
}

}

extern "C" void kernel_launch(int rank, int nranks, ncclComm_t comm,
                              hipStream_t stream,
                              void* const* d_in, const int* in_sizes, int n_in,
                              void* d_out, int out_size,
                              const int* in_dtypes, int out_dtype,
                              void* const* d_out_all,
                              void* const* d_ws_all, size_t ws_bytes) {
    (void)comm; (void)in_dtypes; (void)out_dtype; (void)d_out_all;
    if (nranks != NR || (unsigned)rank >= (unsigned)NR || n_in < 4 || ws_bytes < WS_NEED ||
        in_sizes[0] != MS * KD || in_sizes[1] != KD * NL || out_size != NR * MS * NL) {
        fprintf(stderr, "kernel_launch: unexpected problem shape\n");
        return;
    }
    Params P;
    P.x = (const int8_t*)d_in[0]; P.w = (const int8_t*)d_in[1];
    P.sx = (const float*)d_in[2]; P.sw = (const float*)d_in[3];
    P.out = (float*)d_out;
    P.myws = (char*)d_ws_all[rank];
    for (int j = 0; j < NR; ++j) P.peer[j] = (char*)d_ws_all[(rank + 1 + j) % NR];
    P.rank = rank;
    hipLaunchKernelGGL(ag_gemm_kernel, dim3(GRID), dim3(256), 0, stream, P);
    HIP_CHECK(hipGetLastError());
}
```

```cpp
#include <hip/hip_runtime.h>
#include <rccl/rccl.h>
#include <cstdio>
#include <cstdint>
#include <type_traits>

#define HIP_CHECK(expr) do { hipError_t _e = (expr); if (_e != hipSuccess) { \
    fprintf(stderr, "HIP error %s at %s:%d\n", hipGetErrorString(_e), __FILE__, __LINE__); \
    return; } } while (0)

namespace {

typedef int v2i __attribute__((ext_vector_type(2)));
typedef int v4i __attribute__((ext_vector_type(4)));
typedef unsigned v4u __attribute__((ext_vector_type(4)));
typedef float v4f __attribute__((ext_vector_type(4)));
typedef unsigned long long u64;
#define LAS __attribute__((address_space(3)))
#define GAS __attribute__((address_space(1)))
#define CAS __attribute__((address_space(4)))

constexpr int NR = 8;
constexpr int KD = 4096;
constexpr int NL = 2048;
constexpr int MS = 512;
constexpr int NBANDS = 32;
constexpr unsigned BAND_BYTES = 16 * KD;
constexpr unsigned SHARD_BYTES = MS * KD;
constexpr size_t SLOT_BYTES = (size_t)NR * SHARD_BYTES;
constexpr size_t FLAG_OFF = 2 * SLOT_BYTES;
constexpr size_t CTR_OFF = FLAG_OFF + 16384;
constexpr size_t TMO_OFF = CTR_OFF + 1024 * 64;
constexpr size_t DBG_OFF = TMO_OFF + 4096;
constexpr size_t WS_NEED = DBG_OFF + (1 << 19);
constexpr u64 DEADLINE_TICKS = 40000000ull;

constexpr int NITEMS = 6 + NBANDS;
constexpr int PUSH_SLOTS = 4;
constexpr int GRID = 8 * (PUSH_SLOTS + 28);
#define PUSH_W 16
#define POLL_S 2
#define POLL_AHEAD 5
#define AUX_X 17
#define AUX_OUT 16

struct Params {
    const int8_t* x; const int8_t* w; const float* sx; const float* sw; float* out;
    char* myws;
    char* peer[NR];
    int rank;
};

template <int I, int N, class F> __device__ __forceinline__ void static_for(F&& f) {
    if constexpr (I < N) { f(std::integral_constant<int, I>{}); static_for<I + 1, N>(f); }
}
template <int N> __device__ __forceinline__ void wait_vm() { static_assert(N >= 0 && N < 64, "vmcnt"); asm volatile("s_waitcnt vmcnt(%0)" :: "n"(N) : "memory"); }
__device__ __forceinline__ void wait_lgkm0() { asm volatile("s_waitcnt lgkmcnt(0)" ::: "memory"); }
__device__ __forceinline__ void wg_barrier() { asm volatile("" ::: "memory"); __builtin_amdgcn_s_barrier(); asm volatile("" ::: "memory"); }
#define SCHED_FENCE() __builtin_amdgcn_sched_barrier(0)

__device__ __forceinline__ __amdgpu_buffer_rsrc_t mk_rsrc(const void* p, unsigned bytes) {
    return __builtin_amdgcn_make_buffer_rsrc((void*)p, (short)0, (int)bytes, 0x00020000);
}
__device__ __forceinline__ v4i mk_rsrc_raw(const void* p, unsigned bytes) {
    const u64 a = (u64)p;
    return (v4i){(int)(unsigned)a, (int)((unsigned)(a >> 32) & 0xffffu), (int)bytes, 0x00020000};
}
__device__ __forceinline__ void dma16(v4i rsrc, int voff, int soff, unsigned lds_dst) {
    unsigned keep;
    asm volatile("s_mov_b32 %0, m0\n\ts_mov_b32 m0, %4\n\ts_nop 0\n\tbuffer_load_dwordx4 %1, %2, %3 offen lds\n\ts_mov_b32 m0, %0"
                 : "=&s"(keep) : "v"(voff), "s"(rsrc), "s"(soff), "s"(lds_dst) : "memory");
}

#define DBG_GB(gb) (gb)
#define DBG_WAIT true
#define STAMP(idx) do { } while (0)
#define STAMPC(idx) do { } while (0)

__device__ __forceinline__ void pusher(const int8_t* px, char* myws, int rank, int pj, int q, GAS unsigned* ctr, unsigned call,
                                       LAS char* smem, int lane, int wid) {
    constexpr int W = PUSH_W;
    static_assert(W >= 1 && W <= 16, "window");
    LAS volatile unsigned* done = (LAS volatile unsigned*)(smem + 65536);
    const CAS Params* kp = (const CAS Params*)__builtin_amdgcn_kernarg_segment_ptr();
    char* pws = kp->peer[(unsigned)pj & (NR - 1)];
    if (wid == 0) {
        const unsigned l0 = (unsigned)(size_t)smem;
        const int ss = q * 1024;
        const v4i rs = mk_rsrc_raw(px, SHARD_BYTES);
#define PUSH_DMA(b, r) dma16(rs, (lane ^ (r)) << 4, ss + (b) * (int)BAND_BYTES + (r) * KD, l0 + ((b) & 3) * 16384 + (r) * 1024)
#pragma unroll
        for (int b = 0; b < 3; ++b)
#pragma unroll
            for (int r = 0; r < 16; ++r) PUSH_DMA(b, r);
        if (lane == 0) *done = 0u;
        wait_lgkm0(); wg_barrier();
        const __amdgpu_buffer_rsrc_t rdst = mk_rsrc(pws + (size_t)(call & 1) * SLOT_BYTES + (size_t)rank * SHARD_BYTES, SHARD_BYTES);
        int rd[4];
#pragma unroll
        for (int i = 0; i < 4; ++i) rd[i] = (lane & 15) * 1024 + (((4 * i + (lane >> 4)) ^ (lane & 15)) << 4);
        const int vd = lane * 16, sd = q * 16384;
        v4u ta[16], tb[16];
#define PUSH_READ(t, b) do { _Pragma("unroll") for (int i = 0; i < 16; ++i) t[i] = *(LAS v4u*)(smem + ((b) & 3) * 16384 + rd[i & 3] + (i >> 2) * 256); } while (0)
#define PUSH_BAND(t, tn, b) do { \
            _Pragma("unroll") for (int i = 0; i < 16; ++i) { \
                wait_vm<2 * (W - 1)>(); \
                if (i == W - 1) { if (lane == 0) *done = (unsigned)(b); }        \
                if (i == 8) { PUSH_READ(tn, (b) + 1); }                           \
                PUSH_DMA((b) + 3, i); \
                __builtin_amdgcn_raw_buffer_store_b128(t[i], rdst, vd + 1024 * (i & 3), sd + (b) * (int)BAND_BYTES + 4096 * (i >> 2), 17); \
                SCHED_FENCE(); \
            } } while (0)
        wait_vm<32>();
        PUSH_READ(ta, 0);
        STAMP(2);
#pragma unroll 1
        for (int b = 0; b < NBANDS; b += 2) {
            PUSH_BAND(ta, tb, b);
            PUSH_BAND(tb, ta, b + 1);
        }
        wait_vm<0>();
        if (lane == 0) *done = (unsigned)NBANDS;
#undef PUSH_BAND
#undef PUSH_READ
#undef PUSH_DMA
    } else if (wid == 1) {
        GAS unsigned* flag = (GAS unsigned*)(pws + FLAG_OFF + (size_t)(rank * 4 + q) * 128);
        const unsigned fbase = call * 64u;
        wg_barrier();
        unsigned sent = 0;
        while (sent < (unsigned)NBANDS) {
            const unsigned d = __builtin_amdgcn_readfirstlane(*done);
            if (d > sent) {
                if (lane == 0) __hip_atomic_store(flag, fbase + d, __ATOMIC_RELAXED, __HIP_MEMORY_SCOPE_SYSTEM);
                sent = d;
            } else __builtin_amdgcn_s_sleep(1);
        }
    }
    if (lane == 0) __hip_atomic_store(ctr, call, __ATOMIC_RELAXED, __HIP_MEMORY_SCOPE_AGENT);
    STAMP(1);
}

__device__ __forceinline__ int poll_flags(u64 flaga, unsigned fbase) {
    const unsigned v = __hip_atomic_load((GAS unsigned*)flaga, __ATOMIC_RELAXED, __HIP_MEMORY_SCOPE_SYSTEM);
    return (int)(__builtin_amdgcn_readfirstlane(v) - fbase);
}
__device__ __forceinline__ int wait_flag(u64 flaga, GAS unsigned* tmo, unsigned call, int need, bool& dead, int lane) {
    int known = need;
    if (!dead) {
        const u64 t0 = __builtin_amdgcn_s_memrealtime();
        for (;;) {
            known = poll_flags(flaga, call * 64u);
            if (known >= need) break;
            const u64 el = __builtin_amdgcn_s_memrealtime() - t0;
            if (el > 20000ull) {
                if (el > DEADLINE_TICKS ||
                    __builtin_amdgcn_readfirstlane(__hip_atomic_load(tmo, __ATOMIC_RELAXED, __HIP_MEMORY_SCOPE_AGENT)) == call) {
                    if (lane == 0) __hip_atomic_store(tmo, call, __ATOMIC_RELAXED, __HIP_MEMORY_SCOPE_AGENT);
                    dead = true; break;
                }
                __builtin_amdgcn_s_sleep(8);
            }
            __builtin_amdgcn_s_sleep(1);
        }
    }
    return known;
}

struct Epi {
    LAS char* ex;
    int wr[4];
    int rd;
    int vout;
    float scale;
};

__device__ __forceinline__ v4f silu4(v4i sum, float scale, bool dead) {
    v4f o;
#pragma unroll
    for (int j = 0; j < 4; ++j) {
        const float y = (float)sum[j] * scale;
        const float yc = __builtin_amdgcn_fmed3f(y, -60.0f, 60.0f);
        const float e = __builtin_amdgcn_exp2f(yc * -1.4426950408889634f);
        o[j] = y * __builtin_amdgcn_rcpf(1.0f + e);
    }
    if (dead) o = (v4f){__builtin_nanf(""), __builtin_nanf(""), __builtin_nanf(""), __builtin_nanf("")};
    return o;
}

#define MV(nv) do { __builtin_amdgcn_sched_group_barrier(0x8, 1, 0); __builtin_amdgcn_sched_group_barrier(0x2 | 0x400, nv, 0); } while (0)
__device__ __forceinline__ void band_step(const v4i (&wf)[16][4], v4i (&cur)[16], v4i (&acc)[4], const v4i (&accp)[4],
                                          __amdgpu_buffer_rsrc_t rx, int vrem, int sn,
                                          const Epi& e, int par, __amdgpu_buffer_rsrc_t ro, int sprev, bool dead,
                                          __amdgpu_buffer_rsrc_t rf, int sflag, unsigned& polled) {
    v4i p0, p1, p2, p3, sum;
    v4f y, t, o;
    LAS char* ex = e.ex + par * 4096;
    static_for<0, 16>([&](auto ss) { constexpr int s = ss;
        if constexpr (s == 3) {
            wait_lgkm0(); wg_barrier();
            p0 = *(LAS v4i*)(ex + e.rd); p1 = *(LAS v4i*)(ex + e.rd + 32768);
        }
        if constexpr (s == 4) { p2 = *(LAS v4i*)(ex + e.rd + 65536); p3 = *(LAS v4i*)(ex + e.rd + 98304); }
#pragma unroll
        for (int n = 0; n < 4; ++n) {
            acc[n] = __builtin_amdgcn_mfma_i32_16x16x64_i8(wf[s][n], cur[s], s == 0 ? (v4i){0, 0, 0, 0} : acc[n], 0, 0, 0);
            if constexpr (s < 2) { if (n & 1) { *(LAS v4i*)(ex + e.wr[2 * s + (n >> 1)]) = accp[2 * s + (n >> 1)]; SCHED_FENCE(); } }
        }
        cur[s] = (v4i)__builtin_amdgcn_raw_buffer_load_b128(rx, vrem + 1024 * (s & 3), sn + 4096 * (s >> 2), AUX_X);
        if constexpr (s == POLL_S) polled = __builtin_amdgcn_raw_buffer_load_b32(rf, 0, sflag, 17);
        if constexpr (s == 6) { sum = (p0 + p1) + (p2 + p3); MV(2); MV(2); MV(2); MV(2); }
        if constexpr (s == 7) { for (int j = 0; j < 4; ++j) { y[j] = (float)sum[j]; asm volatile("" : "+v"(y[j])); y[j] *= e.scale; } MV(2); MV(2); MV(2); MV(2); }
        if constexpr (s == 8) { for (int j = 0; j < 4; ++j) t[j] = __builtin_amdgcn_fmed3f(y[j], -60.0f, 60.0f) * -1.4426950408889634f; MV(2); MV(2); MV(2); MV(2); }
        if constexpr (s == 9) { for (int j = 0; j < 4; ++j) t[j] = __builtin_amdgcn_exp2f(t[j]); MV(1); MV(1); MV(1); MV(1); }
        if constexpr (s == 10) { for (int j = 0; j < 4; ++j) { t[j] = 1.0f + t[j]; asm volatile("" : "+v"(t[j])); } MV(1); MV(1); MV(1); MV(1); }
        if constexpr (s == 11) { for (int j = 0; j < 4; ++j) t[j] = __builtin_amdgcn_rcpf(t[j]); MV(1); MV(1); MV(1); MV(1); }
        if constexpr (s == 12) { for (int j = 0; j < 4; ++j) { o[j] = y[j] * t[j]; asm volatile("" : "+v"(o[j])); o[j] = dead ? __builtin_nanf("") : o[j]; } MV(2); MV(2); MV(2); MV(2); }
        if constexpr (s == 13) { __builtin_amdgcn_raw_buffer_store_b128((v4u)o, ro, e.vout, sprev, AUX_OUT); }
        SCHED_FENCE();
    });
}
#undef MV

__global__ void __launch_bounds__(256, 1) ag_gemm_kernel(Params P) {
    __shared__ __attribute__((aligned(16))) char smem[131072];
    const int lane = threadIdx.x & 63;
    const int wid = __builtin_amdgcn_readfirstlane(threadIdx.x >> 6);
    const int xcd = blockIdx.x & 7, slot = blockIdx.x >> 3;
    char* const myws = P.myws;
    const int8_t* const px = P.x;
    const int rank = P.rank;
    STAMP(0);

    GAS unsigned* ctr = (GAS unsigned*)(myws + CTR_OFF + (size_t)(blockIdx.x * 4 + wid) * 64);
    const unsigned call = *(const CAS unsigned*)(u64)ctr + 1u;

    if (slot < PUSH_SLOTS) {
#define PUSH_MAP 0
        pusher(px, myws, rank, (xcd + PUSH_MAP * slot) & 7, slot, ctr, call, (LAS char*)smem, lane, wid);
        return;
    }
    const float sxv = P.sx[0], swv = P.sw[0];

    const int cs = slot - PUSH_SLOTS;
    const int cg = (xcd & 3) * 8 + (cs < 24 ? (cs & 7) : (xcd >> 2) * 4 + (cs - 24));
    const int g = cs < 24 ? (xcd >> 2) * 4 + (cs >> 3) : 3;
    const int src = (rank + 1 + g) & (NR - 1);
    const int n0 = cg * 64;

    v4i wf[16][4];
    v4i xa[16], xb[16];
    LAS char* myl = (LAS char*)smem + wid * 32768;
    {
        const unsigned l0 = (unsigned)(size_t)myl;
        const int row = lane >> 2;
        const v4i rw = mk_rsrc_raw(P.w, KD * NL);
        const int c0 = (lane & 3) ^ ((row >> 2) & 1);
        const int v0 = row * NL + c0 * 16, v1 = row * NL + (c0 ^ 2) * 16;
        const int sw0 = wid * 1024 * NL + n0;
#define DMA_CHUNK(ch, buf) do { _Pragma("unroll") for (int i = 0; i < 16; ++i) \
            dma16(rw, (i & 1) ? v1 : v0, sw0 + ((ch) * 256 + i * 16) * NL, l0 + (buf) * 16384 + i * 1024); } while (0)
        const int gq = lane >> 4, q = (lane & 15) >> 1, p = lane & 1;
        const int f = ((q >> 2) & 1) | ((gq & 1) << 1);
        const int rb = (16 * gq + q) * 64 + 8 * p;
#define READ_CHUNK(ch, buf) do { _Pragma("unroll") for (int s = 0; s < 4; ++s) _Pragma("unroll") for (int n = 0; n < 4; ++n) { \
            LAS char* a = myl + (buf) * 16384 + s * 4096 + rb + 16 * (n ^ f); \
            v2i lo = __builtin_amdgcn_ds_read_tr8_b64_v2i32((LAS v2i*)a); \
            v2i hi = __builtin_amdgcn_ds_read_tr8_b64_v2i32((LAS v2i*)(a + 512)); \
            wf[(ch) * 4 + s][n] = (v4i){lo.x, lo.y, hi.x, hi.y}; } \
            wait_lgkm0(); SCHED_FENCE(); } while (0)
        DMA_CHUNK(0, 0); DMA_CHUNK(1, 1);
        wait_vm<16>(); READ_CHUNK(0, 0);
        DMA_CHUNK(2, 0);
        wait_vm<16>(); READ_CHUNK(1, 1);
        DMA_CHUNK(3, 1);
        wait_vm<16>(); READ_CHUNK(2, 0);
        wait_vm<0>();  READ_CHUNK(3, 1);
#undef DMA_CHUNK
#undef READ_CHUNK
    }
    STAMP(1);

    if (lane == 0) __hip_atomic_store(ctr, call, __ATOMIC_RELAXED, __HIP_MEMORY_SCOPE_AGENT);
    const char* stg = myws + (size_t)(call & 1) * SLOT_BYTES;
    const u64 flagR = (u64)(myws + FLAG_OFF + (size_t)(src * 4 + wid) * 128);
    const u64 flagL = (u64)(myws + FLAG_OFF + (size_t)(rank * 4 + wid) * 128);
    GAS unsigned* tmo = (GAS unsigned*)(myws + TMO_OFF);
    int knownR = 0, knownL = 0;
    bool dead = false;
    constexpr unsigned OUT_BYTES = (unsigned)((size_t)NR * MS * NL * 4);
    const __amdgpu_buffer_rsrc_t ro = mk_rsrc(P.out, OUT_BYTES);
    const __amdgpu_buffer_rsrc_t rx = mk_rsrc(stg, (unsigned)SLOT_BYTES);
    Epi e;
    e.ex = (LAS char*)smem;
    {
        const int r = lane & 15, q = lane >> 4;
#pragma unroll
        for (int n = 0; n < 4; ++n) e.wr[n] = wid * 32768 + r * 256 + 16 * ((4 * n + q) ^ (r & 7));
        const int rrow = 4 * wid + q;
        e.rd = rrow * 256 + 16 * (r ^ (rrow & 7));
        e.vout = (rrow * NL + n0 + 4 * r) * 4;
        e.scale = sxv * swv;
    }
    const int vrem = lane * 16;
    const __amdgpu_buffer_rsrc_t rf = mk_rsrc(myws + FLAG_OFF, 16384);
    const int sflagR = (src * 4 + wid) * 128;
    const unsigned fbase = call * 64u;
    unsigned polled;
#define POLL_OFF(i) ((knownR < NBANDS && knownR < (i) + POLL_AHEAD) ? sflagR : 16384)
#define TAKE_POLL() do { knownR = max(knownR, (int)(__builtin_amdgcn_readfirstlane(polled) - fbase)); } while (0)
    const int nit = g < 2 ? NBANDS + 6 : NBANDS + 4;
    int tab;
    {
        constexpr u64 lm4 = (1ull << 5) | (1ull << 11) | (1ull << 17) | (1ull << 22), lm6 = lm4 | (1ull << 25) | (1ull << 28);
        const u64 lmask = g < 2 ? lm6 : lm4;
        const int nl = __builtin_popcountll(lmask & ((1ull << lane) - 1ull));
        const int bl = nl < 4 ? g + 7 * nl : 24 + 2 * g + nl;
        tab = ((lmask >> lane) & 1ull) ? rank * NBANDS + bl : src * NBANDS + (lane - nl);
        tab = lane < nit ? tab : NR * NBANDS;
    }
#define ITEM_GB(i) __builtin_amdgcn_readlane(tab, (i))
#define ITEM_READY(i, sn) do { \
        const int gb_ = ITEM_GB(i); \
        const bool loc_ = (gb_ >> 5) == rank; \
        const int have_ = loc_ ? knownL : knownR; \
        if (DBG_WAIT && have_ <= (gb_ & 31) && (i) < nit) { \
            const int k_ = wait_flag(loc_ ? flagL : flagR, tmo, call, (gb_ & 31) + 1, dead, lane); \
            if (loc_) knownL = k_; else knownR = k_; \
        } \
        sn = DBG_GB(gb_) * (int)BAND_BYTES + wid * 16384; \
        asm volatile("" ::: "memory"); } while (0)

    {
        int sn;
        ITEM_READY(0, sn);
#pragma unroll
        for (int s = 0; s < 16; ++s) { xa[s] = (v4i)__builtin_amdgcn_raw_buffer_load_b128(rx, vrem + 1024 * (s & 3), sn + 4096 * (s >> 2), AUX_X); SCHED_FENCE(); }
        ITEM_READY(1, sn);
#pragma unroll
        for (int s = 0; s < 16; ++s) { xb[s] = (v4i)__builtin_amdgcn_raw_buffer_load_b128(rx, vrem + 1024 * (s & 3), sn + 4096 * (s >> 2), AUX_X); SCHED_FENCE(); }
    }
    STAMP(60); STAMPC(61);
    int sprev = (int)OUT_BYTES;
    v4i acca[4], accb[4];
#pragma unroll
    for (int n = 0; n < 4; ++n) accb[n] = (v4i){0, 0, 0, 0};
#pragma unroll 1
    for (int i = 0; i < nit; i += 2) {
        int sn;
        ITEM_READY(i + 2, sn);
        band_step(wf, xa, acca, accb, rx, vrem, sn, e, 1, ro, sprev, dead, rf, POLL_OFF(i), polled);
        TAKE_POLL();
        sprev = ITEM_GB(i) * (16 * NL * 4);
        STAMP(2 + i);
        ITEM_READY(i + 3, sn);
        band_step(wf, xb, accb, acca, rx, vrem, sn, e, 0, ro, sprev, dead, rf, POLL_OFF(i + 1), polled);
        TAKE_POLL();
        sprev = ITEM_GB(i + 1) * (16 * NL * 4);
        STAMP(3 + i);
    }
    STAMP(62); STAMPC(63);
#undef ITEM_READY
#undef ITEM_GB
#undef TAKE_POLL
#undef POLL_OFF
    {
        LAS char* ex = e.ex + 4096;
#pragma unroll
        for (int n = 0; n < 4; ++n) *(LAS v4i*)(ex + e.wr[n]) = accb[n];
        wait_lgkm0(); wg_barrier();
        LAS char* ep = ex + e.rd;
        const v4i sum = (*(LAS v4i*)(ep) + *(LAS v4i*)(ep + 32768)) + (*(LAS v4i*)(ep + 65536) + *(LAS v4i*)(ep + 98304));
        __builtin_amdgcn_raw_buffer_store_b128((v4u)silu4(sum, e.scale, dead), ro, e.vout, sprev, AUX_OUT);
        STAMP(2 + NITEMS);
    }
}

}

extern "C" void kernel_launch(int rank, int nranks, ncclComm_t comm,
                              hipStream_t stream,
                              void* const* d_in, const int* in_sizes, int n_in,
                              void* d_out, int out_size,
                              const int* in_dtypes, int out_dtype,
                              void* const* d_out_all,
                              void* const* d_ws_all, size_t ws_bytes) {
    (void)comm; (void)in_dtypes; (void)out_dtype; (void)d_out_all;
    if (nranks != NR || (unsigned)rank >= (unsigned)NR || n_in < 4 || ws_bytes < WS_NEED ||
        in_sizes[0] != MS * KD || in_sizes[1] != KD * NL || out_size != NR * MS * NL) {
        fprintf(stderr, "kernel_launch: unexpected problem shape\n");
        return;
    }
    Params P;
    P.x = (const int8_t*)d_in[0]; P.w = (const int8_t*)d_in[1];
    P.sx = (const float*)d_in[2]; P.sw = (const float*)d_in[3];
    P.out = (float*)d_out;
    P.myws = (char*)d_ws_all[rank];
    for (int j = 0; j < NR; ++j) P.peer[j] = (char*)d_ws_all[(rank + 1 + j) % NR];
    P.rank = rank;
    hipLaunchKernelGGL(ag_gemm_kernel, dim3(GRID), dim3(256), 0, stream, P);
    HIP_CHECK(hipGetLastError());
}
```

```cpp
#include <hip/hip_runtime.h>
#include <rccl/rccl.h>
#include <cstdio>
#include <cstdint>
#include <type_traits>

#define HIP_CHECK(expr) do { hipError_t _e = (expr); if (_e != hipSuccess) { \
    fprintf(stderr, "HIP error %s at %s:%d\n", hipGetErrorString(_e), __FILE__, __LINE__); \
    return; } } while (0)

namespace {

typedef int v2i __attribute__((ext_vector_type(2)));
typedef int v4i __attribute__((ext_vector_type(4)));
typedef unsigned v2u __attribute__((ext_vector_type(2)));
typedef unsigned v4u __attribute__((ext_vector_type(4)));
typedef float v4f __attribute__((ext_vector_type(4)));
typedef unsigned long long u64;
#define LAS __attribute__((address_space(3)))
#define GAS __attribute__((address_space(1)))
#define CAS __attribute__((address_space(4)))

constexpr int NR = 8;
constexpr int KD = 4096;
constexpr int NL = 2048;
constexpr int MS = 512;
constexpr int NBANDS = 32;
constexpr unsigned BAND_BYTES = 16 * KD;
constexpr unsigned SHARD_BYTES = MS * KD;
constexpr size_t SLOT_BYTES = (size_t)NR * SHARD_BYTES;
constexpr size_t FLAG_OFF = 2 * SLOT_BYTES;
constexpr size_t CTR_OFF = FLAG_OFF + 16384;
constexpr size_t TMO_OFF = CTR_OFF + 1024 * 64;
constexpr size_t DBG_OFF = TMO_OFF + 4096;
constexpr size_t WS_NEED = DBG_OFF + (1 << 19);
constexpr u64 DEADLINE_TICKS = 40000000ull;

constexpr int NITEMS = 6 + NBANDS;
constexpr int PUSH_SLOTS = 4;
constexpr int GRID = 8 * (PUSH_SLOTS + 28);
constexpr int WREG = 40960;
#define PUSH_W 16
#define POLL_S 3
#define POLL_SLEEP 1
#define TAIL 32
#define POLL_AHEAD 5
#define EP_BAR 12
#define AUX_X 17
#define AUX_OUT 16

struct Params {
    const int8_t* x; const int8_t* w; const float* sx; const float* sw; float* out;
    char* myws;
    char* peer[NR];
    int rank;
};

template <int I, int N, class F> __device__ __forceinline__ void static_for(F&& f) {
    if constexpr (I < N) { f(std::integral_constant<int, I>{}); static_for<I + 1, N>(f); }
}
template <int N> __device__ __forceinline__ void wait_vm() { static_assert(N >= 0 && N < 64, "vmcnt"); asm volatile("s_waitcnt vmcnt(%0)" :: "n"(N) : "memory"); }
__device__ __forceinline__ void wait_lgkm0() { asm volatile("s_waitcnt lgkmcnt(0)" ::: "memory"); }
__device__ __forceinline__ void wg_barrier() { asm volatile("" ::: "memory"); __builtin_amdgcn_s_barrier(); asm volatile("" ::: "memory"); }
#define SCHED_FENCE() __builtin_amdgcn_sched_barrier(0)

__device__ __forceinline__ __amdgpu_buffer_rsrc_t mk_rsrc(const void* p, unsigned bytes) {
    return __builtin_amdgcn_make_buffer_rsrc((void*)p, (short)0, (int)bytes, 0x00020000);
}
__device__ __forceinline__ v4i mk_rsrc_raw(const void* p, unsigned bytes) {
    const u64 a = (u64)p;
    return (v4i){(int)(unsigned)a, (int)((unsigned)(a >> 32) & 0xffffu), (int)bytes, 0x00020000};
}
__device__ __forceinline__ void dma16(v4i rsrc, int voff, int soff, unsigned lds_dst) {
    unsigned keep;
    asm volatile("s_mov_b32 %0, m0\n\ts_mov_b32 m0, %4\n\ts_nop 0\n\tbuffer_load_dwordx4 %1, %2, %3 offen lds\n\ts_mov_b32 m0, %0"
                 : "=&s"(keep) : "v"(voff), "s"(rsrc), "s"(soff), "s"(lds_dst) : "memory");
}

template <int OFF> __device__ __forceinline__ void ldx(v4i& dst, v4i rsrc, int voff, int soff) {
    asm volatile("buffer_load_dwordx4 %0, %1, %2, %3 offen offset:%4 sc0 sc1" : "+a"(dst) : "v"(voff), "s"(rsrc), "s"(soff), "n"(OFF) : "memory");
}
template <int OFF> __device__ __forceinline__ void ldsx(v4i& dst, unsigned addr) {
    asm volatile("ds_read_b128 %0, %1 offset:%2" : "+a"(dst) : "v"(addr), "n"(OFF) : "memory");
}
template <int OFF> __device__ __forceinline__ void ldsx0(v4i& dst, unsigned addr) {
    asm volatile("ds_read_b128 %0, %1 offset:%2" : "=a"(dst) : "v"(addr), "n"(OFF) : "memory");
}
__device__ __forceinline__ void wait_l16(v4i (&x)[16]) {
    asm volatile("s_waitcnt lgkmcnt(0)" : "+a"(x[0]), "+a"(x[1]), "+a"(x[2]), "+a"(x[3]), "+a"(x[4]), "+a"(x[5]), "+a"(x[6]), "+a"(x[7]),
                 "+a"(x[8]), "+a"(x[9]), "+a"(x[10]), "+a"(x[11]), "+a"(x[12]), "+a"(x[13]), "+a"(x[14]), "+a"(x[15]) :: "memory");
}
template <int N> __device__ __forceinline__ void wait_x1(v4i& a) { asm volatile("s_waitcnt vmcnt(%1)" : "+a"(a) : "n"(N) : "memory"); }
template <int N> __device__ __forceinline__ void wait_x16(v4i (&x)[16]) {
    asm volatile("s_waitcnt vmcnt(%16)" : "+a"(x[0]), "+a"(x[1]), "+a"(x[2]), "+a"(x[3]), "+a"(x[4]), "+a"(x[5]), "+a"(x[6]), "+a"(x[7]),
                 "+a"(x[8]), "+a"(x[9]), "+a"(x[10]), "+a"(x[11]), "+a"(x[12]), "+a"(x[13]), "+a"(x[14]), "+a"(x[15]) : "n"(N) : "memory");
}
#define DMA_UNROLL _Pragma("unroll")
#define DBG_WAIT true
#define STAMP(idx) do { } while (0)
#define STAMPC(idx) do { } while (0)

__device__ __forceinline__ void pusher(const int8_t* px, char* myws, int rank, int pj, int q, GAS unsigned* ctr, unsigned call,
                                       LAS char* smem, int lane, int wid) {
    constexpr int W = PUSH_W;
    static_assert(W >= 1 && W <= 16, "window");
    LAS volatile unsigned* done = (LAS volatile unsigned*)(smem + 65536);
    const CAS Params* kp = (const CAS Params*)__builtin_amdgcn_kernarg_segment_ptr();
    char* pws = kp->peer[(unsigned)pj & (NR - 1)];
    GAS unsigned* flag = (GAS unsigned*)(pws + FLAG_OFF + (size_t)(rank * 4 + q) * 128);
    const unsigned fbase = call * 64u;
    if (wid == 0) {
        const unsigned l0 = (unsigned)(size_t)smem;
        const int ss = q * 1024;
        const v4i rs = mk_rsrc_raw(px, SHARD_BYTES);
#define PUSH_DMA(b, r) dma16(rs, (lane ^ (r)) << 4, ss + (b) * (int)BAND_BYTES + (r) * KD, l0 + ((b) & 3) * 16384 + (r) * 1024)
        DMA_UNROLL
        for (int i = 0; i < 48; ++i) PUSH_DMA(i >> 4, i & 15);
        wg_barrier();
        const __amdgpu_buffer_rsrc_t rdst = mk_rsrc(pws + (size_t)(call & 1) * SLOT_BYTES + (size_t)rank * SHARD_BYTES, SHARD_BYTES);
        int rd[4];
#pragma unroll
        for (int i = 0; i < 4; ++i) rd[i] = (lane & 15) * 1024 + (((4 * i + (lane >> 4)) ^ (lane & 15)) << 4);
        const int vd = lane * 16, sd = q * 16384;
        v4u ta[16], tb[16];
#define PUSH_READ(t, b) do { _Pragma("unroll") for (int i = 0; i < 16; ++i) t[i] = *(LAS v4u*)(smem + ((b) & 3) * 16384 + rd[i & 3] + (i >> 2) * 256); } while (0)
#define PUSH_BAND(t, tn, b) do { \
            _Pragma("unroll") for (int i = 0; i < 16; ++i) { \
                wait_vm<2 * (W - 1)>(); \
                if (i == W - 1) { if (lane == 0) *done = (unsigned)(b); }        \
                if (i == 8) { PUSH_READ(tn, (b) + 1); }                           \
                PUSH_DMA((b) + 3, i); \
                __builtin_amdgcn_raw_buffer_store_b128(t[i], rdst, vd + 1024 * (i & 3), sd + (b) * (int)BAND_BYTES + 4096 * (i >> 2), 17); \
                SCHED_FENCE(); \
            } } while (0)
        wait_vm<32>();
        PUSH_READ(ta, 0);
        STAMP(2);
#pragma unroll 1
        for (int b = 0; b < NBANDS; b += 2) {
            PUSH_BAND(ta, tb, b);
            PUSH_BAND(tb, ta, b + 1);
        }
        wait_vm<0>();
        if (lane == 0) __hip_atomic_store(flag + 1, fbase + NBANDS, __ATOMIC_RELAXED, __HIP_MEMORY_SCOPE_SYSTEM);
#undef PUSH_BAND
#undef PUSH_READ
#undef PUSH_DMA
    } else {
        if (lane == 0) *done = 0u;
        wait_lgkm0(); wg_barrier();
        unsigned sent = 0;
        while (sent < (unsigned)NBANDS - 1u) {
            const unsigned d = __builtin_amdgcn_readfirstlane(*done);
            if (d > sent) {
                if (lane == 0) __hip_atomic_store(flag, fbase + d, __ATOMIC_RELAXED, __HIP_MEMORY_SCOPE_SYSTEM);
                sent = d;
            } else __builtin_amdgcn_s_sleep(1);
        }
    }
    if (lane == 0) __hip_atomic_store(ctr, call, __ATOMIC_RELAXED, __HIP_MEMORY_SCOPE_AGENT);
    STAMP(1);
}

__device__ __forceinline__ int smax(int a, int b) { int r; asm("s_max_i32 %0, %1, %2" : "=s"(r) : "s"(a), "s"(b) : "scc"); return r; }
__device__ __forceinline__ int flags_to_bands(v2u v, unsigned fbase) {
    return smax((int)(__builtin_amdgcn_readfirstlane(v[0]) - fbase), (int)(__builtin_amdgcn_readfirstlane(v[1]) - fbase));
}
constexpr int GAVE_UP = 1 << 30;
__device__ __forceinline__ int wait_flag(GAS u64* flagp, GAS unsigned* tmo, unsigned call, int need, int lane) {
    const u64 t0 = __builtin_amdgcn_s_memrealtime();
    for (;;) {
        const u64 v = __hip_atomic_load(flagp, __ATOMIC_RELAXED, __HIP_MEMORY_SCOPE_SYSTEM);
        const int known = flags_to_bands((v2u){(unsigned)v, (unsigned)(v >> 32)}, call * 64u);
        if (known >= need) return known;
        const u64 el = __builtin_amdgcn_s_memrealtime() - t0;
        if (el > 20000ull) {
            if (el > DEADLINE_TICKS ||
                __builtin_amdgcn_readfirstlane(__hip_atomic_load(tmo, __ATOMIC_RELAXED, __HIP_MEMORY_SCOPE_AGENT)) == call) {
                if (lane == 0) __hip_atomic_store(tmo, call, __ATOMIC_RELAXED, __HIP_MEMORY_SCOPE_AGENT);
                return GAVE_UP;
            }
            __builtin_amdgcn_s_sleep(8);
        }
        __builtin_amdgcn_s_sleep(POLL_SLEEP);
    }
}

struct Epi {
    LAS char* ex;
    int wr[4];
    int rd;
    int vout;
};

constexpr float CLIP2 = 86.56170245333781f;
__device__ __forceinline__ v4f silu4(v4i sum, float scale, float nscl2) {
    v4f o;
#pragma unroll
    for (int j = 0; j < 4; ++j) {
        const float f = (float)sum[j];
        const float e = __builtin_amdgcn_exp2f(__builtin_amdgcn_fmed3f(f * nscl2, -CLIP2, CLIP2));
        o[j] = (f * scale) * __builtin_amdgcn_rcpf(1.0f + e);
    }
    return o;
}

template <class Pre, class Refill>
__device__ __forceinline__ void band_step(const v4i (&wf)[16][4], v4i (&cur)[16], v4i (&acc)[4], const v4i (&accp)[4], Pre&& pre, Refill&& refill,
                                          const Epi& e, int par, __amdgpu_buffer_rsrc_t ro, int sprev, float scale, float nscl2) {
    v4i p0, p1, p2, p3;
    int sum[4];
    float y[4], t[4];
#define PIN(v) asm volatile("" : "+v"(v))
    LAS char* ex = e.ex + par * 4096;
    static_for<0, 64>([&](auto mm) { constexpr int m = mm, s = m >> 2, n = m & 3;
        if constexpr (m == EP_BAR) { wait_lgkm0(); wg_barrier(); }
        if constexpr (n == 0) pre(std::integral_constant<int, s>{});
        acc[n] = __builtin_amdgcn_mfma_i32_16x16x64_i8(wf[s][n], cur[s], s == 0 ? (v4i){0, 0, 0, 0} : acc[n], 0, 0, 0);
        SCHED_FENCE();
        if constexpr (n == 0 && s > 0) refill(std::integral_constant<int, s - 1>{});
        if constexpr (m == 63) refill(std::integral_constant<int, 15>{});
        if constexpr (m == 0) *(LAS v4i*)(ex + e.wr[0]) = accp[0];
        if constexpr (m == 2) *(LAS v4i*)(ex + e.wr[1]) = accp[1];
        if constexpr (m == 5) *(LAS v4i*)(ex + e.wr[2]) = accp[2];
        if constexpr (m == 7) *(LAS v4i*)(ex + e.wr[3]) = accp[3];
        if constexpr (m == EP_BAR + 1) p0 = *(LAS v4i*)(ex + e.rd);
        if constexpr (m == EP_BAR + 2) p1 = *(LAS v4i*)(ex + e.rd + WREG);
        if constexpr (m == EP_BAR + 3) p2 = *(LAS v4i*)(ex + e.rd + 2 * WREG);
        if constexpr (m == EP_BAR + 5) p3 = *(LAS v4i*)(ex + e.rd + 3 * WREG);
        constexpr int k = (s - 6) * 3 + (n - 1);
        if constexpr (s >= 6 && n > 0) {
            if constexpr (k < 2)       { for (int j = 2 * k; j < 2 * k + 2; ++j) { sum[j] = p0[j] + p1[j]; PIN(sum[j]); } }
            else if constexpr (k < 4)  { for (int j = 2 * k - 4; j < 2 * k - 2; ++j) { sum[j] = sum[j] + p2[j] + p3[j]; PIN(sum[j]); } }
            else if constexpr (k < 6)  { for (int j = 2 * k - 8; j < 2 * k - 6; ++j) { y[j] = (float)sum[j]; PIN(y[j]); } }
            else if constexpr (k < 8)  { for (int j = 2 * k - 12; j < 2 * k - 10; ++j) { t[j] = y[j] * nscl2; PIN(t[j]); } }
            else if constexpr (k < 10) { for (int j = 2 * k - 16; j < 2 * k - 14; ++j) { t[j] = __builtin_amdgcn_fmed3f(t[j], -CLIP2, CLIP2); PIN(t[j]); } }
            else if constexpr (k < 14) { t[k - 10] = __builtin_amdgcn_exp2f(t[k - 10]); PIN(t[k - 10]); }
            else if constexpr (k < 16) { for (int j = 2 * k - 28; j < 2 * k - 26; ++j) { t[j] = 1.0f + t[j]; PIN(t[j]); } }
            else if constexpr (k < 20) { t[k - 16] = __builtin_amdgcn_rcpf(t[k - 16]); PIN(t[k - 16]); }
            else if constexpr (k < 22) { for (int j = 2 * k - 40; j < 2 * k - 38; ++j) { y[j] = y[j] * scale; PIN(y[j]); } }
            else if constexpr (k < 24) { for (int j = 2 * k - 44; j < 2 * k - 42; ++j) { y[j] = y[j] * t[j]; PIN(y[j]); } }
            else if constexpr (k == 25) __builtin_amdgcn_raw_buffer_store_b128((v4u)(v4f){y[0], y[1], y[2], y[3]}, ro, e.vout, sprev, AUX_OUT);
        }
        SCHED_FENCE();
    });
#undef PIN
}

__global__ void __launch_bounds__(256, 1) ag_gemm_kernel(Params P) {
    __shared__ __attribute__((aligned(16))) char smem[4 * WREG];
    const int lane = threadIdx.x & 63;
    const int wid = __builtin_amdgcn_readfirstlane(threadIdx.x >> 6);
    const int xcd = blockIdx.x & 7, slot = blockIdx.x >> 3;
    char* const myws = P.myws;
    const int8_t* const px = P.x;
    const int rank = P.rank;
    STAMP(0); STAMPC(50);

    GAS unsigned* ctr = (GAS unsigned*)(myws + CTR_OFF + (size_t)(blockIdx.x * 4 + wid) * 64);

    if (slot < PUSH_SLOTS) {
        if (xcd < NR - 1 && wid < 2)
            pusher(px, myws, rank, xcd, slot, ctr, *(const CAS unsigned*)(u64)ctr + 1u, (LAS char*)smem, lane, wid);
        return;
    }
    const unsigned call = *(const CAS unsigned*)(u64)ctr + 1u;
    const float sxv = P.sx[0], swv = P.sw[0];

    const int cs = slot - PUSH_SLOTS;
    const int cg = (xcd & 3) * 8 + (cs < 24 ? (cs & 7) : (xcd >> 2) * 4 + (cs - 24));
    const int g = cs < 24 ? (xcd >> 2) * 4 + (cs >> 3) : 3;
    const int src = (rank + 1 + g) & (NR - 1);
    const int n0 = cg * 64;
    const int rot = (g + cg) & 3;
    const bool extra = g < 4;
#define LBAND(j) (g + 7 * (j))

    v4i wf[16][4];
    v4i xa[16], xb[16];
    LAS char* myl = (LAS char*)smem + wid * WREG;
    const unsigned l0 = (unsigned)(size_t)myl;
    const v4i rxl = mk_rsrc_raw(px, SHARD_BYTES);
    const int lrot = (lane + 16 * rot) & 63;
#define DMA_LOCAL(j, buf) do { const int sb_ = LBAND(j) * (int)BAND_BYTES + wid * 1024; \
        DMA_UNROLL for (int r = 0; r < 16; ++r) dma16(rxl, (lrot ^ r) << 4, sb_ + r * KD, l0 + (buf) * 16384 + r * 1024); } while (0)
    {
        const int row = lane >> 2;
        const v4i rw = mk_rsrc_raw(P.w, KD * NL);
        const int c0 = (lane & 3) ^ ((row >> 2) & 1);
        const int v0 = row * NL + c0 * 16, v1 = row * NL + (c0 ^ 2) * 16;
        const int sw0 = wid * 1024 * NL + n0;
#define DMA_CHUNK(ch, buf) do { const int sb_ = sw0 + (((ch) + rot) & 3) * 256 * NL; \
            DMA_UNROLL for (int i = 0; i < 16; i += 2) { \
                dma16(rw, v0, sb_ + i * 16 * NL, l0 + (buf) * 16384 + i * 1024); \
                dma16(rw, v1, sb_ + (i + 1) * 16 * NL, l0 + (buf) * 16384 + (i + 1) * 1024); } } while (0)
        const int gq = lane >> 4, q = (lane & 15) >> 1, p = lane & 1;
        const int f = ((q >> 2) & 1) | ((gq & 1) << 1);
        const int rb = (16 * gq + q) * 64 + 8 * p;
#define READ_CHUNK(ch, buf) do { _Pragma("unroll") for (int s = 0; s < 4; ++s) _Pragma("unroll") for (int n = 0; n < 4; ++n) { \
            LAS char* a = myl + (buf) * 16384 + s * 4096 + rb + 16 * (n ^ f); \
            v2i lo = __builtin_amdgcn_ds_read_tr8_b64_v2i32((LAS v2i*)a); \
            v2i hi = __builtin_amdgcn_ds_read_tr8_b64_v2i32((LAS v2i*)(a + 512)); \
            wf[(ch) * 4 + s][n] = (v4i){lo.x, lo.y, hi.x, hi.y}; } \
            wait_lgkm0(); SCHED_FENCE(); } while (0)
        unsigned tch0, tch1;
        {
            const int j0 = lane >> 4, j1 = 4 + (lane >> 4);
            const int ro_ = (2 * (cg & 7) + ((lane >> 3) & 1)) * KD + wid * 1024 + (lane & 7) * 128;
            const int o0 = LBAND(j0) * (int)BAND_BYTES + ro_;
            const int o1 = LBAND(j1) * (int)BAND_BYTES + ro_;
            asm volatile("buffer_load_dword %0, %1, %2, 0 offen" : "=v"(tch0) : "v"(o0), "s"(rxl) : "memory");
            asm volatile("buffer_load_dword %0, %1, %2, 0 offen" : "=v"(tch1) : "v"(o1), "s"(rxl) : "memory");
        }
        DMA_CHUNK(0, 0); DMA_CHUNK(1, 1);
        wait_vm<16>();
        asm volatile("" :: "v"(tch0), "v"(tch1));
        READ_CHUNK(0, 0);
        DMA_CHUNK(2, 0);
        wait_vm<16>(); READ_CHUNK(1, 1);
        DMA_CHUNK(3, 1);
        wait_vm<16>(); READ_CHUNK(2, 0);
        DMA_LOCAL(extra ? 4 : 0, 0);
        wait_vm<16>(); READ_CHUNK(3, 1);
        DMA_LOCAL(1, 1);
#undef DMA_CHUNK
#undef READ_CHUNK
    }
    STAMP(1); STAMPC(51);

    float scale = sxv * swv;
    const float nscl2 = scale * -1.4426950408889634f;
#define SCALE scale
    constexpr unsigned OUT_BYTES = (unsigned)((size_t)NR * MS * NL * 4);
    constexpr int BAND_OUT = 16 * NL * 4;
    const __amdgpu_buffer_rsrc_t ro = mk_rsrc(P.out, OUT_BYTES);
    Epi e;
    e.ex = (LAS char*)smem + 32768;
    {
        const int r = lane & 15, q = lane >> 4;
#pragma unroll
        for (int n = 0; n < 4; ++n) e.wr[n] = wid * WREG + r * 256 + 16 * ((4 * n + q) ^ (r & 7));
        const int rrow = 4 * wid + q;
        e.rd = rrow * 256 + 16 * (r ^ (rrow & 7));
        e.vout = (rrow * NL + n0 + 4 * r) * 4;
    }
    const v4i rx = mk_rsrc_raw(myws + (size_t)(call & 1) * SLOT_BYTES + (size_t)src * SHARD_BYTES, SHARD_BYTES);
    GAS u64* flagp = (GAS u64*)(myws + FLAG_OFF + (size_t)(src * 4 + wid) * 128);
    GAS unsigned* tmo = (GAS unsigned*)(myws + TMO_OFF);
    const unsigned fbase = call * 64u;
    const int vrem = lane * 16;
    int blk[4];
#pragma unroll
    for (int c = 0; c < 4; ++c) blk[c] = wid * 16384 + 4096 * ((c + rot) & 3);
    int known = 0;
#define NEED(b) do { const int nb_ = min((b), NBANDS - 1); \
        if (DBG_WAIT && __builtin_expect(known <= nb_, 0)) { \
            known = wait_flag(flagp, tmo, call, nb_ + 1, lane); \
            if (known == GAVE_UP) scale = __builtin_nanf(""); } \
        asm volatile("" ::: "memory"); } while (0)
#define LOADX(dst, b, s) ldx<1024 * ((s) & 3)>(dst, rx, vrem, (b) * (int)BAND_BYTES + blk[(s) >> 2])

    v4i acca[4], accb[4];
#pragma unroll
    for (int n = 0; n < 4; ++n) accb[n] = (v4i){0, 0, 0, 0};
    int sprev = (int)OUT_BYTES;
    const int sloc = rank * NBANDS * BAND_OUT;

    {
        unsigned lrd[4];
#pragma unroll
        for (int i = 0; i < 4; ++i) lrd[i] = l0 + (lane & 15) * 1024 + (((4 * i + (lane >> 4)) ^ (lane & 15)) << 4);
#define READL(dst, buf, s) ldsx<(buf) * 16384 + 256 * ((s) >> 2)>(dst, lrd[(s) & 3])
        wait_vm<16>();
        static_for<0, 16>([&](auto ss) { constexpr int s = ss; ldsx0<256 * (s >> 2)>(xa[s], lrd[s & 3]); });
        wait_l16(xa);
        DMA_LOCAL(extra ? 0 : 2, 0);
        wait_vm<16>();
        static_for<0, 16>([&](auto ss) { constexpr int s = ss; ldsx0<16384 + 256 * (s >> 2)>(xb[s], lrd[s & 3]); });
        wait_l16(xb);
        DMA_LOCAL(3, 1);
        STAMP(60); STAMPC(61);
#pragma unroll 1
        for (int it = extra ? 0 : 1; it < 2; ++it) {
            band_step(wf, xa, acca, accb, [](auto) {}, [&](auto ss) { constexpr int s = ss;
                if constexpr (s == 8) wait_vm<0>();
                if constexpr (s >= 8) { READL(xa[s - 8], 0, s - 8); READL(xa[s], 0, s); } }, e, 1, ro, sprev, SCALE, nscl2);
            wait_l16(xa);
            if (it == 0) {
                sprev = sloc + LBAND(4) * BAND_OUT;
#pragma unroll
                for (int n = 0; n < 4; ++n) accb[n] = acca[n];
                DMA_LOCAL(2, 0);
            }
        }
        sprev = sloc + LBAND(0) * BAND_OUT;
        STAMP(2);
        band_step(wf, xb, accb, acca, [](auto) {}, [&](auto ss) { constexpr int s = ss;
            if constexpr (s >= 8) { READL(xb[s - 8], 1, s - 8); READL(xb[s], 1, s); } }, e, 0, ro, sprev, SCALE, nscl2);
        wait_l16(xb);
        sprev = sloc + LBAND(1) * BAND_OUT;
        STAMP(3);
#undef READL
    }
#undef DMA_LOCAL
    if (lane == 0) __hip_atomic_store(ctr, call, __ATOMIC_RELAXED, __HIP_MEMORY_SCOPE_AGENT);

    int snext = sloc + LBAND(2) * BAND_OUT, snext2 = sloc + LBAND(3) * BAND_OUT;
    v2u polled;
    const v4i rf = mk_rsrc_raw(myws + FLAG_OFF, 16384);
    const int sflag = (src * 4 + wid) * 128;
#define POLL_ADDR(b) (known < min((b) + POLL_AHEAD, NBANDS) ? sflag : 16384)
#define PRE(xx) [&](auto ss) { if constexpr (ss == 0) wait_x16<18>(xx); }
#define REFILL(xx, bb, pa) [&](auto ss) { constexpr int s = ss; LOADX(xx[s], bb, s); \
        if constexpr (s == POLL_S) asm volatile("buffer_load_dwordx2 %0, off, %1, %2 sc0 sc1" : "=v"(polled) : "s"(rf), "s"(pa) : "memory"); \
        if constexpr (s == 14) { asm volatile("s_waitcnt vmcnt(%1)" : "+v"(polled) : "n"(14 - POLL_S + 1) : "memory"); \
            known = smax(known, flags_to_bands(polled, fbase)); } }
#pragma unroll 1
    for (int b = -2; b < TAIL; b += 2) {
        NEED(b + 2);
        { const int pa = POLL_ADDR(b); band_step(wf, xa, acca, accb, PRE(xa), REFILL(xa, b + 2, pa), e, 1, ro, sprev, SCALE, nscl2); }
        sprev = snext;
        STAMP(8 + b);
        const int b3 = b + 2 < TAIL ? b + 3 : NBANDS;
        NEED(min(b + 3, TAIL));
        { const int pa = POLL_ADDR(b + 1); band_step(wf, xb, accb, acca, PRE(xb), REFILL(xb, b3, pa), e, 0, ro, sprev, SCALE, nscl2); }
        sprev = snext2;
        snext = (src * NBANDS + b + 2) * BAND_OUT; snext2 = snext + BAND_OUT;
        STAMP(9 + b);
        if ((b & 7) == 6) STAMPC(52 + ((b + 2) >> 3));
    }
#define PRE1(xx) [&](auto ss) { constexpr int s = ss; wait_x1<16 - s + 2 * (s < 1 ? 0 : (s > 9 ? 8 : s - 1)) + (s == 15 ? 1 : 0)>(xx[s]); }
#define REFILL1(xx, bb) [&](auto ss) { constexpr int s = ss; \
        if constexpr (s < 8) { LOADX(xx[2 * s], bb, 2 * s); LOADX(xx[2 * s + 1], bb, 2 * s + 1); } }
#pragma unroll 1
    for (int b = TAIL; b < NBANDS; b += 2) {
        NEED(b + 1);
        band_step(wf, xa, acca, accb, PRE1(xa), REFILL1(xb, b + 1), e, 1, ro, sprev, SCALE, nscl2);
        sprev = (src * NBANDS + b) * BAND_OUT;
        NEED(b + 2);
        band_step(wf, xb, accb, acca, PRE1(xb), REFILL1(xa, b + 2), e, 0, ro, sprev, SCALE, nscl2);
        sprev += BAND_OUT;
    }
    STAMP(39); STAMPC(56);
#undef REFILL1
#undef PRE1
#undef PRE
#undef REFILL
#undef LBAND
    STAMP(62); STAMPC(63);
#undef POLL_ADDR
#undef LOADX
#undef NEED
    {
        LAS char* ex = e.ex + 4096;
#pragma unroll
        for (int n = 0; n < 4; ++n) *(LAS v4i*)(ex + e.wr[n]) = accb[n];
        wait_lgkm0(); wg_barrier();
        LAS char* ep = ex + e.rd;
        const v4i sum = (*(LAS v4i*)(ep) + *(LAS v4i*)(ep + WREG)) + (*(LAS v4i*)(ep + 2 * WREG) + *(LAS v4i*)(ep + 3 * WREG));
        __builtin_amdgcn_raw_buffer_store_b128((v4u)silu4(sum, SCALE, nscl2), ro, e.vout, sprev, AUX_OUT);
        STAMP(2 + NITEMS);
    }
    wait_x16<0>(xa); wait_x16<0>(xb);
}

}

extern "C" void kernel_launch(int rank, int nranks, ncclComm_t comm,
                              hipStream_t stream,
                              void* const* d_in, const int* in_sizes, int n_in,
                              void* d_out, int out_size,
                              const int* in_dtypes, int out_dtype,
                              void* const* d_out_all,
                              void* const* d_ws_all, size_t ws_bytes) {
    (void)comm; (void)in_dtypes; (void)out_dtype; (void)d_out_all;
    if (nranks != NR || (unsigned)rank >= (unsigned)NR || n_in < 4 || ws_bytes < WS_NEED ||
        in_sizes[0] != MS * KD || in_sizes[1] != KD * NL || out_size != NR * MS * NL) {
        fprintf(stderr, "kernel_launch: unexpected problem shape\n");
        return;
    }
    Params P;
    P.x = (const int8_t*)d_in[0]; P.w = (const int8_t*)d_in[1];
    P.sx = (const float*)d_in[2]; P.sw = (const float*)d_in[3];
    P.out = (float*)d_out;
    P.myws = (char*)d_ws_all[rank];
    for (int j = 0; j < NR; ++j) P.peer[j] = (char*)d_ws_all[(rank + 1 + j) % NR];
    P.rank = rank;
    hipLaunchKernelGGL(ag_gemm_kernel, dim3(GRID), dim3(256), 0, stream, P);
    HIP_CHECK(hipGetLastError());
}
```

```cpp
#include <hip/hip_runtime.h>
#include <rccl/rccl.h>
#include <cstdio>
#include <cstdint>
#include <type_traits>

#define HIP_CHECK(expr) do { hipError_t _e = (expr); if (_e != hipSuccess) { \
    fprintf(stderr, "HIP error %s at %s:%d\n", hipGetErrorString(_e), __FILE__, __LINE__); \
    return; } } while (0)

namespace {

typedef int v2i __attribute__((ext_vector_type(2)));
typedef int v4i __attribute__((ext_vector_type(4)));
typedef unsigned v2u __attribute__((ext_vector_type(2)));
typedef unsigned v4u __attribute__((ext_vector_type(4)));
typedef float v4f __attribute__((ext_vector_type(4)));
typedef unsigned long long u64;
#define LAS __attribute__((address_space(3)))
#define GAS __attribute__((address_space(1)))
#define CAS __attribute__((address_space(4)))

constexpr int NR = 8;
constexpr int KD = 4096;
constexpr int NL = 2048;
constexpr int MS = 512;
constexpr int NBANDS = 32;
constexpr unsigned BAND_BYTES = 16 * KD;
constexpr unsigned SHARD_BYTES = MS * KD;
constexpr size_t SLOT_BYTES = (size_t)NR * SHARD_BYTES;
constexpr size_t FLAG_OFF = 2 * SLOT_BYTES;
constexpr size_t CTR_OFF = FLAG_OFF + 16384;
constexpr size_t TMO_OFF = CTR_OFF + 1024 * 64;
constexpr size_t DBG_OFF = TMO_OFF + 4096;
constexpr size_t WS_NEED = DBG_OFF + (1 << 19);
constexpr u64 DEADLINE_TICKS = 40000000ull;

constexpr int NITEMS = 6 + NBANDS;
constexpr int PUSH_SLOTS = 4;
constexpr int GRID = 8 * (PUSH_SLOTS + 28);
constexpr int WREG = 40960;
#define PUSH_W 16
#define POLL_S 3
#define START_SLEEP 0
#define POLL_SLEEP 1
#define TAIL 32
#define POLL_AHEAD 5
#define EP_BAR 12
#define AUX_X 17
#define AUX_OUT 16

struct Params {
    const int8_t* x; const int8_t* w; const float* sx; const float* sw; float* out;
    char* myws;
    char* peer[NR];
    int rank;
};

template <int I, int N, class F> __device__ __forceinline__ void static_for(F&& f) {
    if constexpr (I < N) { f(std::integral_constant<int, I>{}); static_for<I + 1, N>(f); }
}
template <int N> __device__ __forceinline__ void wait_vm() { static_assert(N >= 0 && N < 64, "vmcnt"); asm volatile("s_waitcnt vmcnt(%0)" :: "n"(N) : "memory"); }
__device__ __forceinline__ void wait_lgkm0() { asm volatile("s_waitcnt lgkmcnt(0)" ::: "memory"); }
__device__ __forceinline__ void wg_barrier() { asm volatile("" ::: "memory"); __builtin_amdgcn_s_barrier(); asm volatile("" ::: "memory"); }
#define SCHED_FENCE() __builtin_amdgcn_sched_barrier(0)

__device__ __forceinline__ __amdgpu_buffer_rsrc_t mk_rsrc(const void* p, unsigned bytes) {
    return __builtin_amdgcn_make_buffer_rsrc((void*)p, (short)0, (int)bytes, 0x00020000);
}
__device__ __forceinline__ v4i mk_rsrc_raw(const void* p, unsigned bytes) {
    const u64 a = (u64)p;
    return (v4i){(int)(unsigned)a, (int)((unsigned)(a >> 32) & 0xffffu), (int)bytes, 0x00020000};
}
__device__ __forceinline__ void dma16(v4i rsrc, int voff, int soff, unsigned lds_dst) {
    unsigned keep;
    asm volatile("s_mov_b32 %0, m0\n\ts_mov_b32 m0, %4\n\ts_nop 0\n\tbuffer_load_dwordx4 %1, %2, %3 offen lds\n\ts_mov_b32 m0, %0"
                 : "=&s"(keep) : "v"(voff), "s"(rsrc), "s"(soff), "s"(lds_dst) : "memory");
}

template <int OFF> __device__ __forceinline__ void ldx(v4i& dst, v4i rsrc, int voff, int soff) {
    asm volatile("buffer_load_dwordx4 %0, %1, %2, %3 offen offset:%4 sc0 sc1" : "+a"(dst) : "v"(voff), "s"(rsrc), "s"(soff), "n"(OFF) : "memory");
}
template <int OFF> __device__ __forceinline__ void ldsx(v4i& dst, unsigned addr) {
    asm volatile("ds_read_b128 %0, %1 offset:%2" : "+a"(dst) : "v"(addr), "n"(OFF) : "memory");
}
template <int OFF> __device__ __forceinline__ void ldsx0(v4i& dst, unsigned addr) {
    asm volatile("ds_read_b128 %0, %1 offset:%2" : "=a"(dst) : "v"(addr), "n"(OFF) : "memory");
}
__device__ __forceinline__ void wait_l16(v4i (&x)[16]) {
    asm volatile("s_waitcnt lgkmcnt(0)" : "+a"(x[0]), "+a"(x[1]), "+a"(x[2]), "+a"(x[3]), "+a"(x[4]), "+a"(x[5]), "+a"(x[6]), "+a"(x[7]),
                 "+a"(x[8]), "+a"(x[9]), "+a"(x[10]), "+a"(x[11]), "+a"(x[12]), "+a"(x[13]), "+a"(x[14]), "+a"(x[15]) :: "memory");
}
template <int N> __device__ __forceinline__ void wait_x1(v4i& a) { asm volatile("s_waitcnt vmcnt(%1)" : "+a"(a) : "n"(N) : "memory"); }
template <int N> __device__ __forceinline__ void wait_x16(v4i (&x)[16]) {
    asm volatile("s_waitcnt vmcnt(%16)" : "+a"(x[0]), "+a"(x[1]), "+a"(x[2]), "+a"(x[3]), "+a"(x[4]), "+a"(x[5]), "+a"(x[6]), "+a"(x[7]),
                 "+a"(x[8]), "+a"(x[9]), "+a"(x[10]), "+a"(x[11]), "+a"(x[12]), "+a"(x[13]), "+a"(x[14]), "+a"(x[15]) : "n"(N) : "memory");
}
#define DMA_UNROLL _Pragma("unroll")
#define DBG_WAIT true
#define STAMP(idx) do { } while (0)
#define STAMPC(idx) do { } while (0)

__device__ __forceinline__ void pusher(const int8_t* px, char* myws, int rank, int pj, int q, GAS unsigned* ctr, unsigned call,
                                       LAS char* smem, int lane, int wid) {
    constexpr int W = PUSH_W;
    static_assert(W >= 1 && W <= 16, "window");
    LAS volatile unsigned* done = (LAS volatile unsigned*)(smem + 65536);
    const CAS Params* kp = (const CAS Params*)__builtin_amdgcn_kernarg_segment_ptr();
    char* pws = kp->peer[(unsigned)pj & (NR - 1)];
    GAS unsigned* flag = (GAS unsigned*)(pws + FLAG_OFF + (size_t)(rank * 4 + q) * 128);
    const unsigned fbase = call * 64u;
    if (wid == 0) {
        const unsigned l0 = (unsigned)(size_t)smem;
        const int ss = q * 1024;
        const v4i rs = mk_rsrc_raw(px, SHARD_BYTES);
#define PUSH_DMA(b, r) dma16(rs, (lane ^ (r)) << 4, ss + (b) * (int)BAND_BYTES + (r) * KD, l0 + ((b) & 3) * 16384 + (r) * 1024)
        DMA_UNROLL
        for (int i = 0; i < 48; ++i) PUSH_DMA(i >> 4, i & 15);
        wg_barrier();
        const __amdgpu_buffer_rsrc_t rdst = mk_rsrc(pws + (size_t)(call & 1) * SLOT_BYTES + (size_t)rank * SHARD_BYTES, SHARD_BYTES);
        int rd[4];
#pragma unroll
        for (int i = 0; i < 4; ++i) rd[i] = (lane & 15) * 1024 + (((4 * i + (lane >> 4)) ^ (lane & 15)) << 4);
        const int vd = lane * 16, sd = q * 16384;
        v4u ta[16], tb[16];
#define PUSH_READ(t, b) do { _Pragma("unroll") for (int i = 0; i < 16; ++i) t[i] = *(LAS v4u*)(smem + ((b) & 3) * 16384 + rd[i & 3] + (i >> 2) * 256); } while (0)
#define PUSH_BAND(t, tn, b) do { \
            _Pragma("unroll") for (int i = 0; i < 16; ++i) { \
                wait_vm<2 * (W - 1)>(); \
                if (i == W - 1) { if (lane == 0) *done = (unsigned)(b); }        \
                if (i == 8) { PUSH_READ(tn, (b) + 1); }                           \
                PUSH_DMA((b) + 3, i); \
                __builtin_amdgcn_raw_buffer_store_b128(t[i], rdst, vd + 1024 * (i & 3), sd + (b) * (int)BAND_BYTES + 4096 * (i >> 2), 17); \
                SCHED_FENCE(); \
            } } while (0)
        wait_vm<32>();
        PUSH_READ(ta, 0);
        STAMP(2);
#pragma unroll 1
        for (int b = 0; b < NBANDS; b += 2) {
            PUSH_BAND(ta, tb, b);
            PUSH_BAND(tb, ta, b + 1);
        }
        wait_vm<0>();
        if (lane == 0) __hip_atomic_store(flag + 1, fbase + NBANDS, __ATOMIC_RELAXED, __HIP_MEMORY_SCOPE_SYSTEM);
#undef PUSH_BAND
#undef PUSH_READ
#undef PUSH_DMA
    } else {
        if (lane == 0) *done = 0u;
        wait_lgkm0(); wg_barrier();
        unsigned sent = 0;
        while (sent < (unsigned)NBANDS - 1u) {
            const unsigned d = __builtin_amdgcn_readfirstlane(*done);
            if (d > sent) {
                if (lane == 0) __hip_atomic_store(flag, fbase + d, __ATOMIC_RELAXED, __HIP_MEMORY_SCOPE_SYSTEM);
                STAMP(8 + d);
                sent = d;
            } else __builtin_amdgcn_s_sleep(1);
        }
    }
    if (lane == 0) __hip_atomic_store(ctr, call, __ATOMIC_RELAXED, __HIP_MEMORY_SCOPE_AGENT);
    STAMP(1);
}

__device__ __forceinline__ int smax(int a, int b) { int r; asm("s_max_i32 %0, %1, %2" : "=s"(r) : "s"(a), "s"(b) : "scc"); return r; }
__device__ __forceinline__ int flags_to_bands(v2u v, unsigned fbase) {
    return smax((int)(__builtin_amdgcn_readfirstlane(v[0]) - fbase), (int)(__builtin_amdgcn_readfirstlane(v[1]) - fbase));
}
constexpr int GAVE_UP = 1 << 30;
__device__ __forceinline__ int wait_flag(GAS u64* flagp, GAS unsigned* tmo, unsigned call, int need, int lane) {
    const u64 t0 = __builtin_amdgcn_s_memrealtime();
    for (;;) {
        const u64 v = __hip_atomic_load(flagp, __ATOMIC_RELAXED, __HIP_MEMORY_SCOPE_SYSTEM);
        const int known = flags_to_bands((v2u){(unsigned)v, (unsigned)(v >> 32)}, call * 64u);
        if (known >= need) return known;
        const u64 el = __builtin_amdgcn_s_memrealtime() - t0;
        if (el > 20000ull) {
            if (el > DEADLINE_TICKS ||
                __builtin_amdgcn_readfirstlane(__hip_atomic_load(tmo, __ATOMIC_RELAXED, __HIP_MEMORY_SCOPE_AGENT)) == call) {
                if (lane == 0) __hip_atomic_store(tmo, call, __ATOMIC_RELAXED, __HIP_MEMORY_SCOPE_AGENT);
                return GAVE_UP;
            }
            __builtin_amdgcn_s_sleep(8);
        }
        __builtin_amdgcn_s_sleep(POLL_SLEEP);
    }
}

struct Epi {
    LAS char* ex;
    int wr[4];
    int rd;
    int vout;
};

constexpr float CLIP2 = 86.56170245333781f;
__device__ __forceinline__ v4f silu4(v4i sum, float scale, float nscl2) {
    v4f o;
#pragma unroll
    for (int j = 0; j < 4; ++j) {
        const float f = (float)sum[j];
        const float e = __builtin_amdgcn_exp2f(__builtin_amdgcn_fmed3f(f * nscl2, -CLIP2, CLIP2));
        o[j] = (f * scale) * __builtin_amdgcn_rcpf(1.0f + e);
    }
    return o;
}

template <class Pre, class Refill>
__device__ __forceinline__ void band_step(const v4i (&wf)[16][4], v4i (&cur)[16], v4i (&acc)[4], const v4i (&accp)[4], Pre&& pre, Refill&& refill,
                                          const Epi& e, int par, __amdgpu_buffer_rsrc_t ro, int sprev, float scale, float nscl2) {
    v4i p0, p1, p2, p3;
    int sum[4];
    float y[4], t[4];
#define PIN(v) asm volatile("" : "+v"(v))
    LAS char* ex = e.ex + par * 4096;
    static_for<0, 64>([&](auto mm) { constexpr int m = mm, s = m >> 2, n = m & 3;
        if constexpr (m == EP_BAR) { wait_lgkm0(); wg_barrier(); }
        if constexpr (n == 0) pre(std::integral_constant<int, s>{});
        acc[n] = __builtin_amdgcn_mfma_i32_16x16x64_i8(wf[s][n], cur[s], s == 0 ? (v4i){0, 0, 0, 0} : acc[n], 0, 0, 0);
        SCHED_FENCE();
        if constexpr (n == 0 && s > 0) refill(std::integral_constant<int, s - 1>{});
        if constexpr (m == 63) refill(std::integral_constant<int, 15>{});
        if constexpr (m == 0) *(LAS v4i*)(ex + e.wr[0]) = accp[0];
        if constexpr (m == 2) *(LAS v4i*)(ex + e.wr[1]) = accp[1];
        if constexpr (m == 5) *(LAS v4i*)(ex + e.wr[2]) = accp[2];
        if constexpr (m == 7) *(LAS v4i*)(ex + e.wr[3]) = accp[3];
        if constexpr (m == EP_BAR + 1) p0 = *(LAS v4i*)(ex + e.rd);
        if constexpr (m == EP_BAR + 2) p1 = *(LAS v4i*)(ex + e.rd + WREG);
        if constexpr (m == EP_BAR + 3) p2 = *(LAS v4i*)(ex + e.rd + 2 * WREG);
        if constexpr (m == EP_BAR + 5) p3 = *(LAS v4i*)(ex + e.rd + 3 * WREG);
        constexpr int k = (s - 6) * 3 + (n - 1);
        if constexpr (s >= 6 && n > 0) {
            if constexpr (k < 2)       { for (int j = 2 * k; j < 2 * k + 2; ++j) { sum[j] = p0[j] + p1[j]; PIN(sum[j]); } }
            else if constexpr (k < 4)  { for (int j = 2 * k - 4; j < 2 * k - 2; ++j) { sum[j] = sum[j] + p2[j] + p3[j]; PIN(sum[j]); } }
            else if constexpr (k < 6)  { for (int j = 2 * k - 8; j < 2 * k - 6; ++j) { y[j] = (float)sum[j]; PIN(y[j]); } }
            else if constexpr (k < 8)  { for (int j = 2 * k - 12; j < 2 * k - 10; ++j) { t[j] = y[j] * nscl2; PIN(t[j]); } }
            else if constexpr (k < 10) { for (int j = 2 * k - 16; j < 2 * k - 14; ++j) { t[j] = __builtin_amdgcn_fmed3f(t[j], -CLIP2, CLIP2); PIN(t[j]); } }
            else if constexpr (k < 14) { t[k - 10] = __builtin_amdgcn_exp2f(t[k - 10]); PIN(t[k - 10]); }
            else if constexpr (k < 16) { for (int j = 2 * k - 28; j < 2 * k - 26; ++j) { t[j] = 1.0f + t[j]; PIN(t[j]); } }
            else if constexpr (k < 20) { t[k - 16] = __builtin_amdgcn_rcpf(t[k - 16]); PIN(t[k - 16]); }
            else if constexpr (k < 22) { for (int j = 2 * k - 40; j < 2 * k - 38; ++j) { y[j] = y[j] * scale; PIN(y[j]); } }
            else if constexpr (k < 24) { for (int j = 2 * k - 44; j < 2 * k - 42; ++j) { y[j] = y[j] * t[j]; PIN(y[j]); } }
            else if constexpr (k == 25) __builtin_amdgcn_raw_buffer_store_b128((v4u)(v4f){y[0], y[1], y[2], y[3]}, ro, e.vout, sprev, AUX_OUT);
        }
        SCHED_FENCE();
    });
#undef PIN
}

__global__ void __launch_bounds__(256, 1) ag_gemm_kernel(Params P) {
    __shared__ __attribute__((aligned(16))) char smem[4 * WREG];
    const int lane = threadIdx.x & 63;
    const int wid = __builtin_amdgcn_readfirstlane(threadIdx.x >> 6);
    const int xcd = blockIdx.x & 7, slot = blockIdx.x >> 3;
    char* const myws = P.myws;
    const int8_t* const px = P.x;
    const int rank = P.rank;
    STAMP(0); STAMPC(50);

    GAS unsigned* ctr = (GAS unsigned*)(myws + CTR_OFF + (size_t)(blockIdx.x * 4 + wid) * 64);

    if (slot < PUSH_SLOTS) {
        if (xcd < NR - 1 && wid < 2)
            pusher(px, myws, rank, xcd, slot, ctr, *(const CAS unsigned*)(u64)ctr + 1u, (LAS char*)smem, lane, wid);
        return;
    }
    const unsigned call = *(const CAS unsigned*)(u64)ctr + 1u;
    const float sxv = P.sx[0], swv = P.sw[0];

    const int cs = slot - PUSH_SLOTS;
    const int cg = (xcd & 3) * 8 + (cs < 24 ? (cs & 7) : (xcd >> 2) * 4 + (cs - 24));
    const int g = cs < 24 ? (xcd >> 2) * 4 + (cs >> 3) : 3;
    const int src = (rank + 1 + g) & (NR - 1);
    const int n0 = cg * 64;
    const int rot = (g + cg) & 3;
    const bool extra = g < 4;
#define LBAND(j) (g + 7 * (j))

    v4i wf[16][4];
    v4i xa[16], xb[16];
    v4i acca[4], accb[4];
    LAS char* myl = (LAS char*)smem + wid * WREG;
    const unsigned l0 = (unsigned)(size_t)myl;
    const v4i rxl = mk_rsrc_raw(px, SHARD_BYTES);
    const int lrot = (lane + 16 * rot) & 63;
#define DMA_LOCAL(j, buf) do { const int sb_ = LBAND(j) * (int)BAND_BYTES + wid * 1024; \
        DMA_UNROLL for (int r = 0; r < 16; ++r) dma16(rxl, (lrot ^ r) << 4, sb_ + r * KD, l0 + (buf) * 16384 + r * 1024); } while (0)
    unsigned lrd[4];
#pragma unroll
    for (int i = 0; i < 4; ++i) lrd[i] = l0 + (lane & 15) * 1024 + (((4 * i + (lane >> 4)) ^ (lane & 15)) << 4);
#define READL(dst, buf, s) ldsx<(buf) * 16384 + 256 * ((s) >> 2)>(dst, lrd[(s) & 3])

    {
#pragma unroll
        for (int i = 0; i < START_SLEEP / 16; ++i) __builtin_amdgcn_s_sleep(16);
        if (START_SLEEP % 16) __builtin_amdgcn_s_sleep(START_SLEEP % 16);
        DMA_LOCAL(0, 0); DMA_LOCAL(1, 1);
        unsigned tch;
        {
            const int o = LBAND(2 + (lane >> 4)) * (int)BAND_BYTES + (2 * (cg & 7) + ((lane >> 3) & 1)) * KD + wid * 1024 + (lane & 7) * 128;
            asm volatile("buffer_load_dword %0, %1, %2, 0 offen" : "=v"(tch) : "v"(o), "s"(rxl) : "memory");
        }
        const int row = lane >> 2;
        const v4i rw = mk_rsrc_raw(P.w, KD * NL);
        const int c0 = (lane & 3) ^ ((row >> 2) & 1);
        const int v0 = row * NL + c0 * 16, v1 = row * NL + (c0 ^ 2) * 16;
        const int sw0 = wid * 1024 * NL + n0;
#define DMA_CHUNK(ch, buf) do { const int sb_ = sw0 + (((ch) + rot) & 3) * 256 * NL; \
            DMA_UNROLL for (int i = 0; i < 16; i += 2) { \
                dma16(rw, v0, sb_ + i * 16 * NL, l0 + (buf) * 16384 + i * 1024); \
                dma16(rw, v1, sb_ + (i + 1) * 16 * NL, l0 + (buf) * 16384 + (i + 1) * 1024); } } while (0)
        const int gq = lane >> 4, q = (lane & 15) >> 1, p = lane & 1;
        const int f = ((q >> 2) & 1) | ((gq & 1) << 1);
        const int rb = (16 * gq + q) * 64 + 8 * p;
#define READ_CHUNK(ch, buf) do { _Pragma("unroll") for (int s = 0; s < 4; ++s) _Pragma("unroll") for (int n = 0; n < 4; ++n) { \
            LAS char* a = myl + (buf) * 16384 + s * 4096 + rb + 16 * (n ^ f); \
            v2i lo = __builtin_amdgcn_ds_read_tr8_b64_v2i32((LAS v2i*)a); \
            v2i hi = __builtin_amdgcn_ds_read_tr8_b64_v2i32((LAS v2i*)(a + 512)); \
            wf[(ch) * 4 + s][n] = (v4i){lo.x, lo.y, hi.x, hi.y}; } \
            wait_lgkm0(); SCHED_FENCE(); } while (0)
#define MUL_CHUNK(ch) do { _Pragma("unroll") for (int s = 4 * (ch); s < 4 * (ch) + 4; ++s) _Pragma("unroll") for (int n = 0; n < 4; ++n) { \
            acca[n] = __builtin_amdgcn_mfma_i32_16x16x64_i8(wf[s][n], xa[s], s == 0 ? (v4i){0, 0, 0, 0} : acca[n], 0, 0, 0); \
            accb[n] = __builtin_amdgcn_mfma_i32_16x16x64_i8(wf[s][n], xb[s], s == 0 ? (v4i){0, 0, 0, 0} : accb[n], 0, 0, 0); } \
            SCHED_FENCE(); } while (0)
        wait_vm<17>();
        static_for<0, 16>([&](auto ss) { constexpr int s = ss; ldsx0<256 * (s >> 2)>(xa[s], lrd[s & 3]); });
        wait_l16(xa);
        DMA_CHUNK(0, 0);
        wait_vm<16>();
        asm volatile("" :: "v"(tch));
        static_for<0, 16>([&](auto ss) { constexpr int s = ss; ldsx0<16384 + 256 * (s >> 2)>(xb[s], lrd[s & 3]); });
        wait_l16(xb);
        DMA_CHUNK(1, 1);
        wait_vm<16>(); READ_CHUNK(0, 0);
        DMA_CHUNK(2, 0);
        MUL_CHUNK(0);
        wait_vm<16>(); READ_CHUNK(1, 1);
        DMA_CHUNK(3, 1);
        MUL_CHUNK(1);
        wait_vm<16>(); READ_CHUNK(2, 0);
        DMA_LOCAL(extra ? 4 : 2, 0);
        MUL_CHUNK(2);
        wait_vm<16>(); READ_CHUNK(3, 1);
        DMA_LOCAL(3, 1);
        MUL_CHUNK(3);
#undef MUL_CHUNK
#undef DMA_CHUNK
#undef READ_CHUNK
    }
    STAMP(1); STAMPC(51);

    float scale = sxv * swv;
    const float nscl2 = scale * -1.4426950408889634f;
#define SCALE scale
    constexpr unsigned OUT_BYTES = (unsigned)((size_t)NR * MS * NL * 4);
    constexpr int BAND_OUT = 16 * NL * 4;
    const __amdgpu_buffer_rsrc_t ro = mk_rsrc(P.out, OUT_BYTES);
    Epi e;
    e.ex = (LAS char*)smem + 32768;
    {
        const int r = lane & 15, q = lane >> 4;
#pragma unroll
        for (int n = 0; n < 4; ++n) e.wr[n] = wid * WREG + r * 256 + 16 * ((4 * n + q) ^ (r & 7));
        const int rrow = 4 * wid + q;
        e.rd = rrow * 256 + 16 * (r ^ (rrow & 7));
        e.vout = (rrow * NL + n0 + 4 * r) * 4;
    }
    const v4i rx = mk_rsrc_raw(myws + (size_t)(call & 1) * SLOT_BYTES + (size_t)src * SHARD_BYTES, SHARD_BYTES);
    GAS u64* flagp = (GAS u64*)(myws + FLAG_OFF + (size_t)(src * 4 + wid) * 128);
    GAS unsigned* tmo = (GAS unsigned*)(myws + TMO_OFF);
    const unsigned fbase = call * 64u;
    const int vrem = lane * 16;
    int blk[4];
#pragma unroll
    for (int c = 0; c < 4; ++c) blk[c] = wid * 16384 + 4096 * ((c + rot) & 3);
    int known = 0;
#define NEED(b) do { const int nb_ = min((b), NBANDS - 1); \
        if (DBG_WAIT && __builtin_expect(known <= nb_, 0)) { \
            known = wait_flag(flagp, tmo, call, nb_ + 1, lane); \
            if (known == GAVE_UP) scale = __builtin_nanf(""); } \
        asm volatile("" ::: "memory"); } while (0)
#define LOADX(dst, b, s) ldx<1024 * ((s) & 3)>(dst, rx, vrem, (b) * (int)BAND_BYTES + blk[(s) >> 2])
#define FINISH(acc, par, sout) do { LAS char* ex_ = e.ex + (par) * 4096; \
        _Pragma("unroll") for (int n = 0; n < 4; ++n) *(LAS v4i*)(ex_ + e.wr[n]) = acc[n]; \
        wait_lgkm0(); wg_barrier(); \
        LAS char* ep_ = ex_ + e.rd; \
        const v4i sum_ = (*(LAS v4i*)(ep_) + *(LAS v4i*)(ep_ + WREG)) + (*(LAS v4i*)(ep_ + 2 * WREG) + *(LAS v4i*)(ep_ + 3 * WREG)); \
        __builtin_amdgcn_raw_buffer_store_b128((v4u)silu4(sum_, SCALE, nscl2), ro, e.vout, sout, AUX_OUT); } while (0)
    const int sloc = rank * NBANDS * BAND_OUT;
    v2u polled;
    const v4i rf = mk_rsrc_raw(myws + FLAG_OFF, 16384);
    const int sflag = (src * 4 + wid) * 128;

    asm volatile("buffer_load_dwordx2 %0, off, %1, %2 sc0 sc1" : "=v"(polled) : "s"(rf), "s"(sflag) : "memory");
    wait_vm<17>();
    static_for<0, 16>([&](auto ss) { constexpr int s = ss; READL(xa[s], 0, s); });
    FINISH(acca, extra ? 1 : 0, sloc + LBAND(0) * BAND_OUT);
    wait_l16(xa);
    if (extra) DMA_LOCAL(2, 0);
    STAMP(60); STAMPC(61);
    int sprev = sloc + LBAND(1) * BAND_OUT;
    if (extra) {
        band_step(wf, xa, acca, accb, [](auto) {}, [&](auto ss) { constexpr int s = ss;
            if constexpr (s == 8) wait_vm<0>();
            if constexpr (s >= 8) { READL(xa[s - 8], 0, s - 8); READL(xa[s], 0, s); } }, e, 0, ro, sprev, SCALE, nscl2);
        sprev = sloc + LBAND(4) * BAND_OUT;
#pragma unroll
        for (int n = 0; n < 4; ++n) accb[n] = acca[n];
    }
    asm volatile("s_waitcnt vmcnt(1)" : "+v"(polled) :: "memory");
    known = smax(known, flags_to_bands(polled, fbase));
    static_for<0, 16>([&](auto ss) { constexpr int s = ss; READL(xb[s], 1, s); });
    wait_l16(xa); wait_l16(xb);
    STAMP(3);
#undef READL
#undef DMA_LOCAL
    if (lane == 0) __hip_atomic_store(ctr, call, __ATOMIC_RELAXED, __HIP_MEMORY_SCOPE_AGENT);

    int snext = sloc + LBAND(2) * BAND_OUT, snext2 = sloc + LBAND(3) * BAND_OUT;
#define POLL_ADDR(b) (known < min((b) + POLL_AHEAD, NBANDS) ? sflag : 16384)
#define PRE(xx) [&](auto ss) { if constexpr (ss == 0) wait_x16<18>(xx); }
#define REFILL(xx, bb, pa) [&](auto ss) { constexpr int s = ss; LOADX(xx[s], bb, s); \
        if constexpr (s == POLL_S) asm volatile("buffer_load_dwordx2 %0, off, %1, %2 sc0 sc1" : "=v"(polled) : "s"(rf), "s"(pa) : "memory"); \
        if constexpr (s == 14) { asm volatile("s_waitcnt vmcnt(%1)" : "+v"(polled) : "n"(14 - POLL_S + 1) : "memory"); \
            known = smax(known, flags_to_bands(polled, fbase)); } }
#pragma unroll 1
    for (int b = -2; b < TAIL; b += 2) {
        NEED(b + 2);
        { const int pa = POLL_ADDR(b); band_step(wf, xa, acca, accb, PRE(xa), REFILL(xa, b + 2, pa), e, 1, ro, sprev, SCALE, nscl2); }
        sprev = snext;
        STAMP(8 + b);
        const int b3 = b + 2 < TAIL ? b + 3 : NBANDS;
        NEED(min(b + 3, TAIL));
        { const int pa = POLL_ADDR(b + 1); band_step(wf, xb, accb, acca, PRE(xb), REFILL(xb, b3, pa), e, 0, ro, sprev, SCALE, nscl2); }
        sprev = snext2;
        snext = (src * NBANDS + b + 2) * BAND_OUT; snext2 = snext + BAND_OUT;
        STAMP(9 + b);
        if ((b & 7) == 6) STAMPC(52 + ((b + 2) >> 3));
    }
#define PRE1(xx) [&](auto ss) { constexpr int s = ss; wait_x1<16 - s + 2 * (s < 1 ? 0 : (s > 9 ? 8 : s - 1)) + (s == 15 ? 1 : 0)>(xx[s]); }
#define REFILL1(xx, bb) [&](auto ss) { constexpr int s = ss; \
        if constexpr (s < 8) { LOADX(xx[2 * s], bb, 2 * s); LOADX(xx[2 * s + 1], bb, 2 * s + 1); } }
#pragma unroll 1
    for (int b = TAIL; b < NBANDS; b += 2) {
        NEED(b + 1);
        band_step(wf, xa, acca, accb, PRE1(xa), REFILL1(xb, b + 1), e, 1, ro, sprev, SCALE, nscl2);
        sprev = (src * NBANDS + b) * BAND_OUT;
        NEED(b + 2);
        band_step(wf, xb, accb, acca, PRE1(xb), REFILL1(xa, b + 2), e, 0, ro, sprev, SCALE, nscl2);
        sprev += BAND_OUT;
    }
    STAMP(39); STAMPC(56);
#undef REFILL1
#undef PRE1
#undef PRE
#undef REFILL
#undef LBAND
    STAMP(62); STAMPC(63);
#undef POLL_ADDR
#undef LOADX
#undef NEED
    FINISH(accb, 1, sprev);
    STAMP(2 + NITEMS);
#undef FINISH
    wait_x16<0>(xa); wait_x16<0>(xb);
}

}

extern "C" void kernel_launch(int rank, int nranks, ncclComm_t comm,
                              hipStream_t stream,
                              void* const* d_in, const int* in_sizes, int n_in,
                              void* d_out, int out_size,
                              const int* in_dtypes, int out_dtype,
                              void* const* d_out_all,
                              void* const* d_ws_all, size_t ws_bytes) {
    (void)comm; (void)in_dtypes; (void)out_dtype; (void)d_out_all;
    if (nranks != NR || (unsigned)rank >= (unsigned)NR || n_in < 4 || ws_bytes < WS_NEED ||
        in_sizes[0] != MS * KD || in_sizes[1] != KD * NL || out_size != NR * MS * NL) {
        fprintf(stderr, "kernel_launch: unexpected problem shape\n");
        return;
    }
    Params P;
    P.x = (const int8_t*)d_in[0]; P.w = (const int8_t*)d_in[1];
    P.sx = (const float*)d_in[2]; P.sw = (const float*)d_in[3];
    P.out = (float*)d_out;
    P.myws = (char*)d_ws_all[rank];
    for (int j = 0; j < NR; ++j) P.peer[j] = (char*)d_ws_all[(rank + 1 + j) % NR];
    P.rank = rank;
    hipLaunchKernelGGL(ag_gemm_kernel, dim3(GRID), dim3(256), 0, stream, P);
    HIP_CHECK(hipGetLastError());
}
```
